```python
import numpy as np

kernel_name = "gpu_gemm_rs_m4096_k4096_n4096_f32_gelu"


ABSMAX_RTOL = 0.001

COMPUTE_DTYPE = "float32"
EPILOGUE = "gelu"
IN_QUANT = None
OUT_QUANT = None
IN_DTYPES = {"x": "float32", "w_mat": "float32"}
WORLD_SIZE = 8

def setup_inputs():

    x = np.random.randn(4096, 4096).astype(np.float32)
    w_mat = (np.random.randn(4096, 4096) / np.sqrt(4096)).astype(np.float32)
    return {"x": x, "w_mat": w_mat}

def _epilogue(y):
    c = 0.7978845608028654
    return (0.5 * y * (1.0 + np.tanh(c * (y + 0.044715 * y ** 3)))).astype(np.float32)

def reference(x, w_mat):
    y = (x @ w_mat).astype(np.float32)
    return _epilogue(y)
```

```cpp
#include <hip/hip_runtime.h>
#include <rccl/rccl.h>
#include <cstdio>
#include <cstdint>
#include <type_traits>

#define HIP_CHECK(expr) do { hipError_t _e = (expr); if (_e != hipSuccess) { \
    fprintf(stderr, "HIP error %s at %s:%d\n", hipGetErrorString(_e), __FILE__, __LINE__); \
    return; } } while (0)

typedef _Float16 h8 __attribute__((ext_vector_type(8)));
typedef float f4 __attribute__((ext_vector_type(4)));
typedef unsigned u4 __attribute__((ext_vector_type(4)));
typedef unsigned u2 __attribute__((ext_vector_type(2)));
typedef __amdgpu_buffer_rsrc_t rsrc_t;
typedef unsigned long long u64;
#define LAS __attribute__((address_space(3)))
typedef LAS char* lds_ptr;

namespace {

constexpr int NRANK = 8;
constexpr int NSTEP = 64;
constexpr unsigned PIECE = 8192;
constexpr unsigned SHARD_BYTES = 8u << 20;
constexpr unsigned OFF_CNT = 0;
constexpr unsigned OFF_TMO = 16384;
constexpr unsigned OFF_TICKET = 20480;
constexpr unsigned OFF_BOXFLAG = 32768;
constexpr unsigned OFF_FLAG = 65536;
constexpr unsigned FLAG_PAR = 65536;
constexpr unsigned FLAG_OUT = 16384;
constexpr unsigned OFF_DIAG = 262144;
constexpr unsigned OFF_STAGE = 1u << 20;
constexpr unsigned STAGE_PAR = 48u * 64u * PIECE;
constexpr unsigned OFF_BOX = OFF_STAGE + 2u * STAGE_PAR;
constexpr unsigned WS_USED = OFF_BOX + (1024u + 512u) * PIECE;
constexpr u64 DEADLINE_TICKS = 40000000ull;
constexpr int RS3 = 0x00020000;
constexpr int AUX_SYS = 17;
constexpr int AUX_DEV = 16;
constexpr int NOCT = 64;
#define PUSH_SETS 2
#define PUSH_DEPTH 4
constexpr int SLOT_REM = 8, SLOT_OWN = 24;

constexpr int LDS_RING = 6 * 3 * PIECE;
constexpr int LDS_SNAP = LDS_RING;
constexpr int LDS_BAD = LDS_SNAP + 2048;
constexpr int LDS_TOTAL = LDS_BAD + 16;

struct Params {
  const float* x;
  const float* w;
  char* ws[NRANK];
  float* out[NRANK];
  int rank;
  int nitem[2];
  int pad;
  unsigned order[2][NOCT];
};

__device__ __forceinline__ rsrc_t mk_rsrc(const void* p, unsigned bytes) {
  const u64 a = (u64)p;
  const u64 u = (u64)(unsigned)__builtin_amdgcn_readfirstlane((unsigned)a) |
                ((u64)(unsigned)__builtin_amdgcn_readfirstlane((unsigned)(a >> 32)) << 32);
  return __builtin_amdgcn_make_buffer_rsrc((void*)u, (short)0, (int)bytes, RS3);
}
__device__ __forceinline__ u64 now() { return __builtin_amdgcn_s_memrealtime(); }
#define WAIT_VM0() do { __builtin_amdgcn_s_waitcnt(0x0F70); asm volatile("s_waitcnt vmcnt(0)" ::: "memory"); } while (0)
#define WAIT_LGKM0() asm volatile("s_waitcnt lgkmcnt(0)" ::: "memory")

#define STAMP(idx) do {} while (0)

__device__ __forceinline__ unsigned pk2(float a, float b) {
  typedef _Float16 h2 __attribute__((ext_vector_type(2)));
  h2 v = {(_Float16)a, (_Float16)b};
  return __builtin_bit_cast(unsigned, v);
}

__device__ __forceinline__ float asf(unsigned v) { return __builtin_bit_cast(float, v); }

__device__ __forceinline__ unsigned get_call(const Params& P, int rank, int wv) {
  unsigned* c = (unsigned*)(P.ws[rank] + OFF_CNT) + (blockIdx.x * 8 + wv);
  unsigned v = __hip_atomic_load(c, __ATOMIC_RELAXED, __HIP_MEMORY_SCOPE_AGENT) + 1u;
  v = __builtin_amdgcn_readfirstlane(v);
  __hip_atomic_store(c, v, __ATOMIC_RELAXED, __HIP_MEMORY_SCOPE_AGENT);
  return v;
}

template <int AUX>
__device__ __forceinline__ void store_piece(const u4 (&h)[8], rsrc_t rd, unsigned l16, unsigned so) {
#pragma unroll
  for (int f = 0; f < 8; ++f)
    __builtin_amdgcn_raw_buffer_store_b128(h[f], rd, l16 + (f & 3) * 1024, so + (f >> 2) * 4096, AUX);
  asm volatile("s_nop 1" ::: "memory");
}

__device__ __forceinline__ void load_raw(const Params& P, int box, int lane, u4 (&raw)[16]) {
  const int t = box >> 7, RB = box & 63;
  const bool isw = (box & 64) == 0;
  const rsrc_t rs = mk_rsrc(isw ? (const void*)P.w : (const void*)P.x, SHARD_BYTES);
  const unsigned voff = isw ? (unsigned)(lane >> 4) * 65536u + (unsigned)(lane & 15) * 16u
                            : (unsigned)(lane & 15) * 2048u + (unsigned)(lane >> 4) * 16u;
  const unsigned src = isw ? (unsigned)t * 1048576u + (unsigned)RB * 256u : (unsigned)RB * 131072u + (unsigned)t * 256u;
  const unsigned s1 = isw ? 262144u : 32768u, s0 = isw ? 16384u : 64u;
#pragma unroll
  for (int n = 0; n < 16; ++n)
    raw[n] = __builtin_amdgcn_raw_buffer_load_b128(rs, voff, src + (n >> 2) * s1 + (n & 3) * s0, 0);
}

__device__ __forceinline__ void to_half(int box, const u4 (&raw)[16], u4 (&h)[8]) {
  if ((box & 64) == 0) {
#pragma unroll
    for (int kb = 0; kb < 2; ++kb)
#pragma unroll
      for (int i = 0; i < 4; ++i)
#pragma unroll
        for (int e = 0; e < 4; ++e)
          h[kb * 4 + i][e] = pk2(asf(raw[kb * 8 + 2 * e][i]), asf(raw[kb * 8 + 2 * e + 1][i]));
  } else {
#pragma unroll
    for (int f = 0; f < 8; ++f) {
      h[f][0] = pk2(asf(raw[2 * f][0]), asf(raw[2 * f][1]));
      h[f][1] = pk2(asf(raw[2 * f][2]), asf(raw[2 * f][3]));
      h[f][2] = pk2(asf(raw[2 * f + 1][0]), asf(raw[2 * f + 1][1]));
      h[f][3] = pk2(asf(raw[2 * f + 1][2]), asf(raw[2 * f + 1][3]));
    }
  }
}

__device__ __forceinline__ unsigned stage_off(int B, int t, int src) { return (unsigned)((src * 8 + t) * 48 + B) * PIECE; }
__device__ __forceinline__ unsigned flag_off(int B, int t, int src) { return (unsigned)(B * 64 + t * 8 + src) * 4u; }

__device__ __forceinline__ unsigned convert(const Params& P, int rank, int box, int wv, int lane) {
  u4 raw[16], h[8];
  load_raw(P, box, lane, raw);
  const unsigned call = get_call(P, rank, wv);
  to_half(box, raw, h);
  const int t = box >> 7, RB = box & 63;
  const bool isw = (box & 64) == 0;
  const unsigned par = call & 1u;
  const unsigned stage = OFF_STAGE + par * STAGE_PAR, flagb = OFF_FLAG + par * FLAG_PAR;
  const unsigned tag = call << 8;
  const unsigned l16 = (unsigned)lane * 16u;
  const rsrc_t rl = mk_rsrc(P.ws[rank], WS_USED);
  const int owner = RB >> 3;
  const int helper = ((owner >> 2) ^ 1) * 4 + ((RB & 7) >> 1);
  int selfB = -1;
  if (isw) { if ((RB >> 5) == (rank >> 2)) selfB = 16 + (RB & 31); }
  else if (owner == rank) selfB = RB & 7;
  else if (helper == rank) selfB = 8 + 2 * (owner & 3) + (RB & 1);
  store_piece<AUX_DEV>(h, rl, l16, OFF_BOX + (unsigned)box * PIECE);
  if (selfB >= 0) store_piece<AUX_DEV>(h, rl, l16, stage + stage_off(selfB, t, rank));
  WAIT_VM0();
  if (lane == 0) {
    __builtin_amdgcn_raw_buffer_store_b32(tag, rl, 0, OFF_BOXFLAG + (unsigned)box * 4u, AUX_DEV);
    if (selfB >= 0) __builtin_amdgcn_raw_buffer_store_b32(tag, rl, 0, flagb + flag_off(selfB, t, rank), AUX_DEV);
  }
  return call;
}

struct Desc {
  int box;
  unsigned dst;
  unsigned flag;
  unsigned dep;
  int isout;
};

__device__ __forceinline__ Desc decode(const Params& P, int rank, int dest, int p) {
  const int G = dest >> 2, q = dest & 3;
  const unsigned o = P.order[(rank >> 2) ^ G][p >> 3];
  const int type = o >> 5, t = (o >> 2) & 7, sub = o & 3, u = p & 7;
  Desc d;
  if (type == 3) {
    const int rg = o & 15;
    d.box = 1024 + q * 128 + rg * 8 + u;
    d.dst = (unsigned)((rank & 3) * 128 + rg * 8) * 16384u + (unsigned)((rank >> 2) * 2048 + u * 256) * 4u;
    d.flag = FLAG_OUT + (unsigned)((rank & 3) * 128 + rg * 8 + u) * 4u;
    d.dep = 1024u + (unsigned)(q * 32 + u * 4);
    d.isout = 1;
    return d;
  }
  int B, blk;
  if (type == 0) {
    const int cb = (sub * 8 + u + 4 * rank) & 31;
    blk = G * 32 + cb; B = 16 + cb;
  } else {
    const int v = (u + rank) & 7;
    blk = 64 + ((type == 1) ? 8 * ((G ^ 1) * 4 + (v >> 1)) + 2 * q + (v & 1) : 8 * dest + v);
    B = (type == 1) ? 8 + v : v;
  }
  d.box = t * 128 + blk;
  d.dst = stage_off(B, t, rank);
  d.flag = flag_off(B, t, rank);
  d.dep = (unsigned)d.box;
  d.isout = 0;
  return d;
}

template <int N> __device__ __forceinline__ void wait_vm() { asm volatile("s_waitcnt vmcnt(%0)" :: "i"(N) : "memory"); }
constexpr int AB = 88;
#define ACLOB "a88","a89","a90","a91","a92","a93","a94","a95","a96","a97","a98","a99","a100","a101","a102","a103","a104","a105","a106","a107","a108","a109","a110","a111","a112","a113","a114","a115","a116","a117","a118","a119","a120","a121"
__device__ __forceinline__ unsigned uni(unsigned v) { return __builtin_amdgcn_readfirstlane(v); }
template <int F> __device__ __forceinline__ void a_store(rsrc_t r, unsigned vo, unsigned so) {
  asm volatile("buffer_store_dwordx4 a[%c3:%c4], %0, %1, %2 offen sc0 sc1" :: "v"(vo), "s"(r), "s"(so), "i"(AB + F * 4), "i"(AB + F * 4 + 3) : "memory", ACLOB);
}
template <int F> __device__ __forceinline__ void a_load(rsrc_t r, unsigned vo, unsigned so) {
  asm volatile("buffer_load_dwordx4 a[%c3:%c4], %0, %1, %2 offen sc1" :: "v"(vo), "s"(r), "s"(so), "i"(AB + F * 4), "i"(AB + F * 4 + 3) : "memory", ACLOB);
}
template <int F> __device__ __forceinline__ void a_put(const u4& v) {
  asm volatile("v_accvgpr_write_b32 a[%c4], %0\n\tv_accvgpr_write_b32 a[%c5], %1\n\tv_accvgpr_write_b32 a[%c6], %2\n\tv_accvgpr_write_b32 a[%c7], %3"
               :: "v"(v[0]), "v"(v[1]), "v"(v[2]), "v"(v[3]), "i"(AB + F * 4), "i"(AB + F * 4 + 1), "i"(AB + F * 4 + 2), "i"(AB + F * 4 + 3) : "memory", ACLOB);
}
__device__ __forceinline__ void a_sample(rsrc_t r, unsigned vo, unsigned so) {
  asm volatile("s_nop 4\n\tbuffer_load_dword a120, %0, %1, %2 offen sc1" :: "v"(vo), "s"(r), "s"(so) : "memory", ACLOB);
}
template <int R> __device__ __forceinline__ unsigned a_get() {
  unsigned v;
  asm volatile("v_accvgpr_read_b32 %0, a[%c1]\n\ts_nop 1" : "=v"(v) : "i"(R) : "memory", ACLOB);
  return v;
}
template <int I, int N, class Fn> __device__ __forceinline__ void static_for(Fn&& fn) {
  if constexpr (I < N) { fn(std::integral_constant<int, I>{}); static_for<I + 1, N>(fn); }
}

__device__ __forceinline__ void pusher(const Params& P, int rank, int dd, int sw, int wv, int lane) {
  constexpr int NSW = 8 * PUSH_SETS, D = PUSH_DEPTH;
  const int dest = rank ^ dd;
  const int nitem = P.nitem[dd >> 2];
  const unsigned l16 = (unsigned)lane * 16u;
  const Desc d0 = decode(P, rank, dest, sw), d1 = decode(P, rank, dest, sw + NSW), d2 = decode(P, rank, dest, sw + 2 * NSW);
  u4 ra[16], rb[16], h[8];
  unsigned* cnt_p = (unsigned*)(P.ws[rank] + OFF_CNT) + (blockIdx.x * 8 + wv);
  const unsigned cnt_v = __hip_atomic_load(cnt_p, __ATOMIC_RELAXED, __HIP_MEMORY_SCOPE_AGENT);
  load_raw(P, d0.box, lane, ra);
  load_raw(P, d1.box, lane, rb);
  const unsigned call = uni(cnt_v) + 1u;
  const unsigned par = call & 1u;
  const unsigned stage = OFF_STAGE + par * STAGE_PAR, flagb = OFF_FLAG + par * FLAG_PAR;
  const rsrc_t rl = mk_rsrc(P.ws[rank], WS_USED);
  const rsrc_t rd = mk_rsrc(P.ws[dest], WS_USED);
  const rsrc_t ro = mk_rsrc(P.out[dest], SHARD_BYTES);
  unsigned* tmo = (unsigned*)(P.ws[rank] + OFF_TMO);
  const unsigned ticket = OFF_TICKET + (par * 8u + (unsigned)dd) * 128u;
  const unsigned want = call & 0xffffffu;
  const unsigned dep4 = (unsigned)(lane & 3) * 4u;

  unsigned prev_flag = 0xffffffffu, prev_tag = 0;
  Desc nn = d2; int have_nn = 0;
  auto flush_prev = [&]() {
    if (prev_flag != 0xffffffffu) {
      if (lane == 0) __builtin_amdgcn_raw_buffer_store_b32(prev_tag, rd, 0, uni(flagb + prev_flag), AUX_SYS);
      prev_flag = 0xffffffffu;
    }
  };
  auto step = [&](auto LOAD, const Desc& d, unsigned tag, int ldbox, bool fetch) {
    constexpr int L = decltype(LOAD)::value;
    const rsrc_t r = d.isout ? ro : rd;
    const unsigned so = uni(d.isout ? d.dst : stage + d.dst), st = d.isout ? 16384u : 1024u;
    const unsigned lo = uni(OFF_BOX + (unsigned)ldbox * PIECE);
    if (fetch && lane == 0)
      asm volatile("v_accvgpr_write_b32 a121, 1\n\ts_nop 4\n\tbuffer_atomic_add a121, off, %0, %1 sc0" :: "s"(rl), "s"(ticket) : "memory", ACLOB);
    have_nn = 0;
    asm volatile("s_nop 4" ::: "memory");
    static_for<0, 8>([&](auto FI) {
      constexpr int f = decltype(FI)::value;
      a_store<f>(r, l16, so + f * st);
      if (L == 1) a_load<f>(rl, l16, lo + f * 1024);
      if (L != 2 || f >= D - 1) wait_vm<L == 1 ? 2 * D - 1 : D - 1>();
      if (f == (L == 2 ? D - 1 : D - 2)) flush_prev();
      if (f == 4 && fetch) {
        const int p = 3 * NSW + (int)uni(a_get<121>());
        if (p < nitem) {
          nn = decode(P, rank, dest, p); have_nn = 1;
          a_sample(rl, nn.isout ? dep4 : 0u, uni(OFF_BOXFLAG + nn.dep * 4u));
        }
      }
    });
    prev_flag = d.flag; prev_tag = tag;
  };
  auto put_all = [&]() {
    asm volatile("s_nop 1" ::: "memory");
    static_for<0, 8>([&](auto FI) { a_put<decltype(FI)::value>(h[decltype(FI)::value]); });
  };
  typedef std::integral_constant<int, 0> NO;
  typedef std::integral_constant<int, 1> YES;
  typedef std::integral_constant<int, 2> FIRST;

  to_half(d0.box, ra, h);
  put_all();
  STAMP(1);
  step(FIRST{}, d0, call << 8, 0, false);
  load_raw(P, d2.box, lane, ra);
  __hip_atomic_store(cnt_p, call, __ATOMIC_RELAXED, __HIP_MEMORY_SCOPE_AGENT);
  if (sw == 0) __hip_atomic_store((unsigned*)(P.ws[rank] + (ticket ^ 1024u)), 0u, __ATOMIC_RELAXED, __HIP_MEMORY_SCOPE_AGENT);
  to_half(d1.box, rb, h);
  put_all();
  step(FIRST{}, d1, call << 8, 0, true);
  to_half(d2.box, ra, h);
  put_all();
  wait_vm<3>();
  Desc cur = d2; unsigned tag = call << 8;
  const u64 t0 = now();
  while (true) {
    wait_vm<6>();
    const unsigned fl0 = a_get<120>();
    const Desc nxt = nn; const int have_nxt = have_nn;
    unsigned ntag = uni(fl0);
    const bool ready = have_nxt && __all((fl0 >> 8) == want);
    if (ready) {
      step(YES{}, cur, tag, nxt.box, true);
    } else {
      step(NO{}, cur, tag, 0, have_nxt != 0);
      wait_vm<0>();
      flush_prev();
      if (!have_nxt) break;
      bool fail = false;
      while (true) {
        const unsigned fl = __builtin_amdgcn_raw_buffer_load_b32(rl, nxt.isout ? dep4 : 0u, uni(OFF_BOXFLAG + nxt.dep * 4u), AUX_DEV);
        ntag = uni(fl);
        if (__all((fl >> 8) == want)) break;
        const u64 dt = now() - t0;
        if (dt > 100000ull) {
          const unsigned tv = uni(__hip_atomic_load(tmo, __ATOMIC_RELAXED, __HIP_MEMORY_SCOPE_AGENT));
          if (tv == call || dt > DEADLINE_TICKS) {
            __hip_atomic_store(tmo, call, __ATOMIC_RELAXED, __HIP_MEMORY_SCOPE_AGENT);
            fail = true;
            break;
          }
        }
        asm volatile("" ::: "memory");
        __builtin_amdgcn_s_sleep(2);
      }
      if (fail) break;
      const unsigned lo = uni(OFF_BOX + (unsigned)nxt.box * PIECE);
      asm volatile("s_nop 4" ::: "memory");
      static_for<0, 8>([&](auto FI) { a_load<decltype(FI)::value>(rl, l16, lo + decltype(FI)::value * 1024); });
      if (have_nn) a_sample(rl, nn.isout ? dep4 : 0u, uni(OFF_BOXFLAG + nn.dep * 4u));
      wait_vm<0>();
    }
    cur = nxt; tag = ntag;
  }
  STAMP(11);
}

__device__ __forceinline__ float gelu(float y) {
  const float c2 = -2.0f * 0.7978845608028654f * 1.4426950408889634f;
  const float u = y * (1.0f + 0.044715f * y * y);
  const float e = __builtin_amdgcn_exp2f(c2 * u);
  return y * __builtin_amdgcn_rcpf(1.0f + e);
}

__device__ __forceinline__ void dma4(unsigned m0v, unsigned voff, rsrc_t rs, unsigned soff) {
  asm volatile("s_nop 4\n\ts_mov_b32 m0, %0\n\ts_nop 0\n\t"
               "buffer_load_dwordx4 %1, %2, %3 offen lds\n\t"
               "buffer_load_dwordx4 %1, %2, %3 offen offset:1024 lds\n\t"
               "buffer_load_dwordx4 %1, %2, %3 offen offset:2048 lds\n\t"
               "buffer_load_dwordx4 %1, %2, %3 offen offset:3072 lds"
               :: "s"(m0v), "v"(voff), "s"(rs), "s"(soff) : "memory");
}
__device__ __forceinline__ void dma_poll(unsigned m0v, unsigned voff, rsrc_t rs) {
  asm volatile("s_nop 4\n\ts_mov_b32 m0, %0\n\ts_nop 0\n\t"
               "buffer_load_dword %1, %2, 0 offen sc0 sc1 lds"
               :: "s"(m0v), "v"(voff), "s"(rs) : "memory");
}

__device__ __forceinline__ int step_of(int n, int rotc) {
  return (n & ~7) + ((rotc - n) & 7);
}

template <int NWP>
__device__ __forceinline__ void gemm_tile(const Params& P, int rank, lds_ptr smem, int xb0, int cb0, int dest, int orow0,
                                          bool remote, unsigned call, int wv, int lane) {
  constexpr int NPIECE = 2 + NWP;
  constexpr int STAGE_B = NPIECE * PIECE;
  constexpr int NS = (NWP == 1) ? 6 : 4;
  constexpr int MF = (NWP == 1) ? 1 : 2;
  const bool loader = wv < 2 * NPIECE;
  const int pi = wv >> 1;
  const int myB = pi < 2 ? xb0 + pi : 16 + cb0 + (pi - 2);
  const int rotc = cb0 >> 2;
  const unsigned lds0 = (unsigned)(uintptr_t)smem;
  volatile LAS unsigned* snap = (volatile LAS unsigned*)(smem + LDS_SNAP + wv * 256);
  volatile LAS unsigned* bad = (volatile LAS unsigned*)(smem + LDS_BAD);
  snap[lane] = 0;
  if (threadIdx.x == 0) *bad = 0;
  WAIT_VM0(); WAIT_LGKM0();
  __builtin_amdgcn_s_barrier();
  const unsigned par = call & 1u;
  const unsigned stage = OFF_STAGE + par * STAGE_PAR, flagb = OFF_FLAG + par * FLAG_PAR;
  const rsrc_t rs = mk_rsrc(P.ws[rank], WS_USED);
  const unsigned l16 = (unsigned)lane * 16u;
  const unsigned poll_off = flagb + (unsigned)(myB * 64 + step_of(lane, rotc)) * 4u;
  const unsigned src0 = stage + (unsigned)myB * PIECE + (unsigned)(wv & 1) * 4096u;
  const unsigned dst0 = lds0 + (unsigned)wv * 4096u;
  const unsigned snap_m0 = lds0 + LDS_SNAP + (unsigned)wv * 256u;
  unsigned* tmo = (unsigned*)(P.ws[rank] + OFF_TMO);

  const int wm = (NWP == 1) ? wv : (wv >> 1) * 2;
  const int wn = (NWP == 1) ? 0 : (wv & 1);
  const lds_ptr a_rd = smem + wm * 2048 + lane * 16;
  const lds_ptr b_rd = smem + (2 + wn) * PIECE + lane * 16;

  f4 acc[MF][4];
#pragma unroll
  for (int m = 0; m < MF; ++m)
#pragma unroll
    for (int i = 0; i < 4; ++i) acc[m][i] = f4{0.f, 0.f, 0.f, 0.f};

  int issued = 0, avail = 0, islot = 0, cslot = 0;
  STAMP(1);
  for (int d = 0; d < NSTEP; ++d) {
    if (loader) {
      const int lim = (d + NS - 1 < NSTEP) ? d + NS - 1 : NSTEP;
      if (avail < lim) {
        const u64 m = __ballot((snap[lane] >> 8) == (call & 0xffffffu));
        avail = (~m == 0ull) ? 64 : __builtin_ctzll(~m);
      }
      if (avail <= d) {
        const u64 t0 = now();
        while (true) {
          if (lane >= avail && lane < avail + 16) dma_poll(snap_m0, poll_off, rs);
          WAIT_VM0();
          const u64 m = __ballot((snap[lane] >> 8) == (call & 0xffffffu));
          avail = (~m == 0ull) ? 64 : __builtin_ctzll(~m);
          if (avail > d) break;
          const u64 dt = now() - t0;
          if (dt > 100000ull) {
            const unsigned tv = __builtin_amdgcn_readfirstlane(__hip_atomic_load(tmo, __ATOMIC_RELAXED, __HIP_MEMORY_SCOPE_AGENT));
            if (tv == call || dt > DEADLINE_TICKS) {
              if (lane == 0) { __hip_atomic_store(tmo, call, __ATOMIC_RELAXED, __HIP_MEMORY_SCOPE_AGENT); *bad = 1; }
              snap[lane] = call << 8;
              avail = 64;
              WAIT_VM0(); WAIT_LGKM0();
              break;
            }
          }
          __builtin_amdgcn_s_sleep(4);
        }
      }
      const int upto = avail < lim ? avail : lim;
      while (issued < upto) {
        dma4(dst0 + (unsigned)islot * STAGE_B, l16, rs, src0 + stage_off(0, issued >> 3, (rotc - issued) & 7));
        ++issued;
        islot = (islot + 1 == NS) ? 0 : islot + 1;
      }
      const int ahead = issued - d - 1;
      if (avail < NSTEP && avail <= lim + 2) {
        if (lane >= avail && lane < avail + 16) dma_poll(snap_m0, poll_off, rs);
        switch (ahead) {
          case 0: asm volatile("s_waitcnt vmcnt(1)" ::: "memory"); break;
          case 1: asm volatile("s_waitcnt vmcnt(5)" ::: "memory"); break;
          case 2: asm volatile("s_waitcnt vmcnt(9)" ::: "memory"); break;
          case 3: asm volatile("s_waitcnt vmcnt(13)" ::: "memory"); break;
          default: asm volatile("s_waitcnt vmcnt(17)" ::: "memory"); break;
        }
      } else {
        switch (ahead) {
          case 0: asm volatile("s_waitcnt vmcnt(0)" ::: "memory"); break;
          case 1: asm volatile("s_waitcnt vmcnt(4)" ::: "memory"); break;
          case 2: asm volatile("s_waitcnt vmcnt(8)" ::: "memory"); break;
          case 3: asm volatile("s_waitcnt vmcnt(12)" ::: "memory"); break;
          default: asm volatile("s_waitcnt vmcnt(16)" ::: "memory"); break;
        }
      }
    }
    asm volatile("s_waitcnt lgkmcnt(0)\n\ts_barrier" ::: "memory");
    const int so = cslot * STAGE_B;
    h8 a[MF][2], b[2][4];
#pragma unroll
    for (int kb = 0; kb < 2; ++kb) {
#pragma unroll
      for (int m = 0; m < MF; ++m) a[m][kb] = *(const LAS h8*)(a_rd + so + m * 2048 + kb * 1024);
#pragma unroll
      for (int i = 0; i < 4; ++i) b[kb][i] = *(const LAS h8*)(b_rd + so + (kb * 4 + i) * 1024);
    }
#pragma unroll
    for (int kb = 0; kb < 2; ++kb)
#pragma unroll
      for (int m = 0; m < MF; ++m)
#pragma unroll
        for (int i = 0; i < 4; ++i)
          acc[m][i] = __builtin_amdgcn_mfma_f32_16x16x32_f16(a[m][kb], b[kb][i], acc[m][i], 0, 0, 0);
    cslot = (cslot + 1 == NS) ? 0 : cslot + 1;
  }

  const bool isbad = *bad != 0;
  const unsigned row = (unsigned)(wm * 16 + (lane >> 4) * 4);
  const rsrc_t ro = remote ? rs : mk_rsrc(P.out[rank], SHARD_BYTES);
  const unsigned ooff = remote ? OFF_BOX + (unsigned)(1024 + dest * 128 + (row >> 3) * 8 + (cb0 >> 2)) * PIECE + (row & 7) * 1024u +
                                     (unsigned)((cb0 & 3) * 256 + (lane & 15) * 16)
                               : ((unsigned)orow0 + row) * 16384u + (unsigned)((rank >> 2) * 2048 + (cb0 + wn) * 64 + (lane & 15) * 4) * 4u;
  const unsigned rstride = remote ? 1024u : 16384u;
#pragma unroll
  for (int m = 0; m < MF; ++m)
#pragma unroll
    for (int r = 0; r < 4; ++r) {
      f4 v = {gelu(acc[m][0][r]), gelu(acc[m][1][r]), gelu(acc[m][2][r]), gelu(acc[m][3][r])};
      if (isbad) { const float qn = __builtin_nanf(""); v = f4{qn, qn, qn, qn}; }
      __builtin_amdgcn_raw_buffer_store_b128(__builtin_bit_cast(u4, v), ro, ooff + (unsigned)(m * 16 + r) * rstride, 0, AUX_DEV);
    }
  STAMP(10);
  if (remote) {
    WAIT_VM0();
    __builtin_amdgcn_s_barrier();
    if (threadIdx.x == 0)
      __builtin_amdgcn_raw_buffer_store_b32(call << 8, rs, 0, OFF_BOXFLAG + (unsigned)(1024 + dest * 32 + cb0) * 4u, AUX_DEV);
  }
  STAMP(11);
}

__device__ __forceinline__ void final_wait(const Params& P, int rank, int wv, int lane) {
  const unsigned call = get_call(P, rank, wv);
  const unsigned flagb = OFF_FLAG + (call & 1u) * FLAG_PAR;
  const rsrc_t rl = mk_rsrc(P.ws[rank], WS_USED);
  unsigned* tmo = (unsigned*)(P.ws[rank] + OFF_TMO);
  const unsigned want = call << 8;
  const u64 t0 = now();
  while (true) {
    bool ok = true;
#pragma unroll
    for (int b = 0; b < 2; ++b) {
      const u4 v = __builtin_amdgcn_raw_buffer_load_b128(rl, (unsigned)lane * 16u, flagb + FLAG_OUT + b * 1024, AUX_SYS);
      ok &= v[0] == want && v[1] == want && v[2] == want && v[3] == want;
    }
    if (__all(ok)) break;
    const u64 dt = now() - t0;
    if (dt > 100000ull) {
      const unsigned tv = __builtin_amdgcn_readfirstlane(__hip_atomic_load(tmo, __ATOMIC_RELAXED, __HIP_MEMORY_SCOPE_AGENT));
      if (tv == call || dt > DEADLINE_TICKS) {
        __hip_atomic_store(tmo, call, __ATOMIC_RELAXED, __HIP_MEMORY_SCOPE_AGENT);
        for (int i = lane; i < 512; i += 64)
          P.out[rank][(size_t)((i >> 3) * 8) * 4096 + ((rank >> 2) ^ 1) * 2048 + (i & 7) * 256] = __builtin_nanf("");
        break;
      }
    }
    asm volatile("" ::: "memory");
    __builtin_amdgcn_s_sleep(2);
  }
  STAMP(11);
}

}

__global__ void __launch_bounds__(512, 2) fused_gemm_rs(Params P) {
  __shared__ __attribute__((aligned(1024))) char smem_arr[LDS_TOTAL];
  const lds_ptr smem = (lds_ptr)smem_arr;
  const int wv = __builtin_amdgcn_readfirstlane(threadIdx.x >> 6);
  const int lane = threadIdx.x & 63;
  const int xcd = blockIdx.x & 7, slot = blockIdx.x >> 3;
  const int rank = P.rank & 7;
  STAMP(0);
  if (slot < SLOT_REM) {
    if (wv < 2) {
      const int dd = 1 + (slot & 3) * 2 + wv;
      if (dd < 8) { if ((slot >> 2) < PUSH_SETS) pusher(P, rank, dd, (slot >> 2) * 8 + xcd, wv, lane); }
      else if (blockIdx.x == 24) final_wait(P, rank, wv, lane);
    }
    return;
  }
  const int box = wv * 192 + ((int)blockIdx.x - SLOT_REM * 8);
  const unsigned call = box < 1024 ? convert(P, rank, box, wv, lane) : get_call(P, rank, wv);
  STAMP(12);
  if (slot < SLOT_OWN) {
    const int s = slot - SLOT_REM, a = s >> 2, cb = xcd * 4 + (s & 3);
    gemm_tile<1>(P, rank, smem, 8 + 2 * a, cb, a, 0, true, call, wv, lane);
  } else {
    const int s = slot - SLOT_OWN, a = s >> 1, cb = xcd * 4 + (s & 1) * 2;
    gemm_tile<2>(P, rank, smem, 2 * a, cb, 0, a * 128, false, call, wv, lane);
  }
}

extern "C" void kernel_launch(int rank, int nranks, ncclComm_t comm,
                              hipStream_t stream,
                              void* const* d_in, const int* in_sizes, int n_in,
                              void* d_out, int out_size,
                              const int* in_dtypes, int out_dtype,
                              void* const* d_out_all,
                              void* const* d_ws_all, size_t ws_bytes) {
  (void)comm; (void)in_dtypes; (void)out_dtype; (void)d_out;
  if (nranks != NRANK || n_in != 2 || in_sizes[0] != 4096 * 512 || in_sizes[1] != 512 * 4096 ||
      out_size != 512 * 4096 || ws_bytes < WS_USED || (unsigned)rank >= (unsigned)NRANK) {
    fprintf(stderr, "kernel_launch: unexpected problem shape\n");
    return;
  }
  Params P;
  P.x = (const float*)d_in[0];
  P.w = (const float*)d_in[1];
  for (int p = 0; p < NRANK; ++p) { P.ws[p] = (char*)d_ws_all[p]; P.out[p] = (float*)d_out_all[p]; }
  P.rank = rank; P.pad = 0;
#define SEQ {1, 0}, {0, 0}, {1, 1}, {2, 0}, {1, 2}, {0, 1}, {2, 1}, {1, 3}, {0, 2}, {2, 2}, {1, 4}, {0, 3}, \
            {2, 3}, {1, 5}, {0, 4}, {2, 4}, {1, 6}, {0, 5}, {1, 7}, {0, 6}, {0, 7}, {2, 5}, {2, 6}, {2, 7}
  static const signed char seq[24][2] = {SEQ};
  int n = 0;
  for (int i = 0; i < 24; ++i) {
    const int type = seq[i][0], t = seq[i][1];
    const int cnt = type == 0 ? 4 : 1;
    for (int s = 0; s < cnt; ++s) { P.order[0][n] = P.order[1][n] = (unsigned)(type << 5 | t << 2 | s); ++n; }
  }
  for (int i = 0; i < 16; ++i) { P.order[0][48 + i] = 0; P.order[1][48 + i] = (unsigned)(3 << 5 | i); }
  P.nitem[0] = 384;
  P.nitem[1] = 512;
  fused_gemm_rs<<<256, 512, 0, stream>>>(P);
  HIP_CHECK(hipGetLastError());
}
```

```cpp
#include <hip/hip_runtime.h>
#include <rccl/rccl.h>
#include <cstdio>
#include <cstdint>
#include <cstdlib>
#include <type_traits>

#define HIP_CHECK(expr) do { hipError_t _e = (expr); if (_e != hipSuccess) { \
    fprintf(stderr, "HIP error %s at %s:%d\n", hipGetErrorString(_e), __FILE__, __LINE__); \
    return; } } while (0)

typedef _Float16 h8 __attribute__((ext_vector_type(8)));
typedef float f4 __attribute__((ext_vector_type(4)));
typedef unsigned u4 __attribute__((ext_vector_type(4)));
typedef unsigned u2 __attribute__((ext_vector_type(2)));
typedef __amdgpu_buffer_rsrc_t rsrc_t;
typedef unsigned long long u64;
#define LAS __attribute__((address_space(3)))
typedef LAS char* lds_ptr;

namespace {

constexpr int NRANK = 8;
constexpr int NSTEP = 64;
constexpr unsigned PIECE = 8192;
constexpr unsigned SHARD_BYTES = 8u << 20;
constexpr unsigned OFF_CNT = 0;
constexpr unsigned OFF_TMO = 16384;
constexpr unsigned OFF_TICKET = 20480;
constexpr unsigned OFF_BOXFLAG = 32768;
constexpr unsigned OFF_FLAG = 65536;
constexpr unsigned FLAG_PAR = 65536;
constexpr unsigned FLAG_OUT = 16384;
constexpr unsigned OFF_DIAG = 262144;
constexpr unsigned OFF_STAGE = 1u << 20;
constexpr unsigned STAGE_PAR = 48u * 64u * PIECE;
constexpr unsigned OFF_BOX = OFF_STAGE + 2u * STAGE_PAR;
constexpr unsigned WS_USED = OFF_BOX + (1024u + 512u) * PIECE;
constexpr u64 DEADLINE_TICKS = 40000000ull;
constexpr int RS3 = 0x00020000;
constexpr int AUX_SYS = 17;
constexpr int AUX_DEV = 16;
constexpr int NOCT = 60;
#define DMA_POL ""
#define POLL_AHEAD 1
#define POLL_LATE 8
#define POLL_SLEEP 2
#define PUSH_DEPTH 7
#define NOPEN 4
#define OPEN_DEPTH 2
#define CONV_DELAYS 0x1E141C1810080202ull
constexpr int SLOT_REM = 4, SLOT_OWN = 20;
constexpr int NGEMM = 256 - SLOT_REM * 8;

constexpr int LDS_RING = 6 * 3 * PIECE;
constexpr int LDS_SNAP = LDS_RING;
constexpr int LDS_QUEUE = LDS_SNAP + 1024;
constexpr int LDS_BAD = LDS_QUEUE + 256;
constexpr int LDS_TOTAL = LDS_BAD + 16;

struct Link {
  char* ws;
  float* out;
  u2 list[NOCT];
};
struct Params {
  const float* x;
  const float* w;
  char* ws;
  float* out;
  int rank;
  int nitem[2];
  int pad;
  Link link[7];
};

__device__ __forceinline__ rsrc_t mk_rsrc(const void* p, unsigned bytes) {
  const u64 a = (u64)p;
  const u64 u = (u64)(unsigned)__builtin_amdgcn_readfirstlane((unsigned)a) |
                ((u64)(unsigned)__builtin_amdgcn_readfirstlane((unsigned)(a >> 32)) << 32);
  return __builtin_amdgcn_make_buffer_rsrc((void*)u, (short)0, (int)bytes, RS3);
}
__device__ __forceinline__ u64 now() { return __builtin_amdgcn_s_memrealtime(); }
#define WAIT_VM0() do { __builtin_amdgcn_s_waitcnt(0x0F70); asm volatile("s_waitcnt vmcnt(0)" ::: "memory"); } while (0)
#define WAIT_LGKM0() asm volatile("s_waitcnt lgkmcnt(0)" ::: "memory")

#define STAMP(idx) do {} while (0)

__device__ __forceinline__ unsigned pk2(float a, float b) {
  typedef _Float16 h2 __attribute__((ext_vector_type(2)));
  h2 v = {(_Float16)a, (_Float16)b};
  return __builtin_bit_cast(unsigned, v);
}

__device__ __forceinline__ float asf(unsigned v) { return __builtin_bit_cast(float, v); }

__device__ __forceinline__ unsigned get_call(const Params& P, int rank, int wv) {
  unsigned* c = (unsigned*)(P.ws + OFF_CNT) + (blockIdx.x * 8 + wv);
  unsigned v = __hip_atomic_load(c, __ATOMIC_RELAXED, __HIP_MEMORY_SCOPE_AGENT) + 1u;
  v = __builtin_amdgcn_readfirstlane(v);
  __hip_atomic_store(c, v, __ATOMIC_RELAXED, __HIP_MEMORY_SCOPE_AGENT);
  return v;
}

template <int AUX>
__device__ __forceinline__ void store_piece(const u4 (&h)[8], rsrc_t rd, unsigned l16, unsigned so) {
#pragma unroll
  for (int f = 0; f < 8; ++f)
    __builtin_amdgcn_raw_buffer_store_b128(h[f], rd, l16 + (f & 3) * 1024, so + (f >> 2) * 4096, AUX);
  asm volatile("s_nop 1" ::: "memory");
}

__device__ __forceinline__ void load_raw(const Params& P, int box, int lane, u4 (&raw)[16]) {
  const int t = box >> 7, RB = box & 63;
  const bool isw = (box & 64) == 0;
  const rsrc_t rs = mk_rsrc(isw ? (const void*)P.w : (const void*)P.x, SHARD_BYTES);
  const unsigned voff = isw ? (unsigned)(lane >> 4) * 65536u + (unsigned)(lane & 15) * 16u
                            : (unsigned)(lane & 15) * 2048u + (unsigned)(lane >> 4) * 16u;
  const unsigned src = isw ? (unsigned)t * 1048576u + (unsigned)RB * 256u : (unsigned)RB * 131072u + (unsigned)t * 256u;
  const unsigned s1 = isw ? 262144u : 32768u, s0 = isw ? 16384u : 64u;
#pragma unroll
  for (int n = 0; n < 16; ++n)
    raw[n] = __builtin_amdgcn_raw_buffer_load_b128(rs, voff, src + (n >> 2) * s1 + (n & 3) * s0, 0);
}

__device__ __forceinline__ void to_half(int box, const u4 (&raw)[16], u4 (&h)[8]) {
  if ((box & 64) == 0) {
#pragma unroll
    for (int kb = 0; kb < 2; ++kb)
#pragma unroll
      for (int i = 0; i < 4; ++i)
#pragma unroll
        for (int e = 0; e < 4; ++e)
          h[kb * 4 + i][e] = pk2(asf(raw[kb * 8 + 2 * e][i]), asf(raw[kb * 8 + 2 * e + 1][i]));
  } else {
#pragma unroll
    for (int f = 0; f < 8; ++f) {
      h[f][0] = pk2(asf(raw[2 * f][0]), asf(raw[2 * f][1]));
      h[f][1] = pk2(asf(raw[2 * f][2]), asf(raw[2 * f][3]));
      h[f][2] = pk2(asf(raw[2 * f + 1][0]), asf(raw[2 * f + 1][1]));
      h[f][3] = pk2(asf(raw[2 * f + 1][2]), asf(raw[2 * f + 1][3]));
    }
  }
}

__device__ __forceinline__ unsigned stage_off(int B, int t, int src) { return (unsigned)((src * 8 + t) * 48 + B) * PIECE; }
__device__ __forceinline__ unsigned flag_off(int B, int t, int src) { return (unsigned)(B * 64 + t * 8 + src) * 4u; }

__device__ __forceinline__ unsigned convert(const Params& P, int rank, int box, int wv, int lane) {
  u4 raw[16], h[8];
  load_raw(P, box, lane, raw);
  const unsigned call = get_call(P, rank, wv);
  to_half(box, raw, h);
  const int t = box >> 7, RB = box & 63;
  const bool isw = (box & 64) == 0;
  const unsigned par = call & 1u;
  const unsigned stage = OFF_STAGE + par * STAGE_PAR, flagb = OFF_FLAG + par * FLAG_PAR;
  const unsigned tag = call << 8;
  const unsigned l16 = (unsigned)lane * 16u;
  const rsrc_t rl = mk_rsrc(P.ws, WS_USED);
  const int owner = RB >> 3;
  const int helper = ((owner >> 2) ^ 1) * 4 + ((RB & 7) >> 1);
  int selfB = -1;
  if (isw) { if ((RB >> 5) == (rank >> 2)) selfB = 16 + (RB & 31); }
  else if (owner == rank) selfB = RB & 7;
  else if (helper == rank) selfB = 8 + 2 * (owner & 3) + (RB & 1);
  store_piece<AUX_DEV>(h, rl, l16, OFF_BOX + (unsigned)box * PIECE);
  if (selfB >= 0) store_piece<AUX_DEV>(h, rl, l16, stage + stage_off(selfB, t, rank));
  WAIT_VM0();
  if (lane == 0) {
    __builtin_amdgcn_raw_buffer_store_b32(tag, rl, 0, OFF_BOXFLAG + (unsigned)box * 4u, AUX_DEV);
    if (selfB >= 0) __builtin_amdgcn_raw_buffer_store_b32(tag, rl, 0, flagb + flag_off(selfB, t, rank), AUX_DEV);
  }
  return call;
}

struct Desc {
  unsigned src;
  unsigned dst;
  unsigned flag;
  unsigned dep;
  int isout;
};

__device__ __forceinline__ Desc decode(unsigned lo, unsigned hi, unsigned v, unsigned spar, unsigned fpar) {
  const bool alt = lo & 0x2000u, isout = lo & 0x4000u, fwd = lo & 0x8000u;
  const unsigned w8 = (v >> 1) * 8u + (v & 1u);
  const unsigned si = alt ? w8 : v;
  Desc d;
  d.src = ((lo & 0x1fffu) + si) * PIECE + (fwd ? spar : 0u);
  d.dst = ((lo >> 16) + (isout ? (v >> 1) * 128u + (v & 1u) : v * 8u)) * 1024u + (isout ? 0u : spar);
  d.flag = ((hi & 0xffffu) + (isout ? w8 : v * 64u)) * 4u + OFF_FLAG + fpar;
  d.dep = ((hi >> 16) + (isout ? (v & 1u) * 4u : fwd ? v * 64u : si)) * 4u + (fwd ? fpar : 0u);
  d.isout = isout;
  return d;
}

template <int N> __device__ __forceinline__ void wait_vm() { asm volatile("s_waitcnt vmcnt(%0)" :: "i"(N) : "memory"); }
constexpr int AB = 220;
#define ACLOB "v220","v221","v222","v223","v224","v225","v226","v227","v228","v229","v230","v231","v232","v233","v234","v235","v236","v237","v238","v239","v240","v241","v242","v243","v244","v245","v246","v247","v248","v249","v250","v251","v252","v253","v254"
__device__ __forceinline__ unsigned uni(unsigned v) { return __builtin_amdgcn_readfirstlane(v); }
template <int F> __device__ __forceinline__ void a_store(rsrc_t r, unsigned vo, unsigned so) {
  asm volatile("buffer_store_dwordx4 v[%c3:%c4], %0, %1, %2 offen sc0 sc1" :: "v"(vo), "s"(r), "s"(so), "i"(AB + F * 4), "i"(AB + F * 4 + 3) : "memory", ACLOB);
}
template <int F> __device__ __forceinline__ void a_load(rsrc_t r, unsigned vo, unsigned so) {
  asm volatile("buffer_load_dwordx4 v[%c3:%c4], %0, %1, %2 offen sc1" :: "v"(vo), "s"(r), "s"(so), "i"(AB + F * 4), "i"(AB + F * 4 + 3) : "memory", ACLOB);
}
template <int R> __device__ __forceinline__ void a_sample(rsrc_t r, unsigned vo, unsigned so) {
  asm volatile("s_nop 4\n\tbuffer_load_dword v[%c3], %0, %1, %2 offen sc0 sc1" :: "v"(vo), "s"(r), "s"(so), "i"(R) : "memory", ACLOB);
}
template <int R> __device__ __forceinline__ unsigned a_get() {
  unsigned v;
  asm volatile("v_mov_b32 %0, v[%c1]\n\ts_nop 1" : "=v"(v) : "i"(R) : "memory", ACLOB);
  return v;
}
template <int I, int N, class Fn> __device__ __forceinline__ void static_for(Fn&& fn) {
  if constexpr (I < N) { fn(std::integral_constant<int, I>{}); static_for<I + 1, N>(fn); }
}

__device__ __forceinline__ unsigned starter(const Params& P, int rank, int dd, int p, int wv, int lane) {
  const Link& L = P.link[dd - 1];
  const u2 e = L.list[p >> 3];
  const unsigned v = (unsigned)(p + rank) & 7u;
  const int box = (int)(decode(e[0], e[1], v, 0, 0).src / PIECE) - (int)(OFF_BOX / PIECE);
  u4 raw[16], h[8];
  unsigned* cnt_p = (unsigned*)(P.ws + OFF_CNT) + (blockIdx.x * 8 + wv);
  const unsigned cnt_v = __hip_atomic_load(cnt_p, __ATOMIC_RELAXED, __HIP_MEMORY_SCOPE_AGENT);
  load_raw(P, box, lane, raw);
  const rsrc_t rd = mk_rsrc(L.ws, WS_USED);
  const unsigned call = uni(cnt_v) + 1u;
  const Desc d = decode(e[0], e[1], v, (call & 1u) * STAGE_PAR, (call & 1u) * FLAG_PAR);
  to_half(box, raw, h);
  STAMP(13);
#pragma unroll
  for (int f = 0; f < 8; ++f) {
    __builtin_amdgcn_raw_buffer_store_b128(h[f], rd, (unsigned)lane * 16u + f * 1024, uni(d.dst), AUX_SYS);
    wait_vm<OPEN_DEPTH - 1>();
  }
  __hip_atomic_store(cnt_p, call, __ATOMIC_RELAXED, __HIP_MEMORY_SCOPE_AGENT);
  WAIT_VM0();
  if (lane == 0) __builtin_amdgcn_raw_buffer_store_b32(call << 8, rd, 0, uni(d.flag), AUX_SYS);
  return call;
}

__device__ __forceinline__ void pusher(const Params& P, int rank, int dd, int sw, int wv, int lane) {
  constexpr int D = PUSH_DEPTH;
  const Link& L = P.link[dd - 1];
  const unsigned l16 = (unsigned)lane * 16u;
  const u2 ent = L.list[lane < NOCT ? lane : 0];
  const unsigned call = get_call(P, rank, wv);
  const int nitem = P.nitem[dd >> 2];
  const rsrc_t rl = mk_rsrc(P.ws, WS_USED);
  const rsrc_t rd = mk_rsrc(L.ws, WS_USED);
  const rsrc_t ro = mk_rsrc(L.out, SHARD_BYTES);
  const unsigned par = call & 1u;
  const unsigned spar = par * STAGE_PAR, fpar = par * FLAG_PAR;
  unsigned* tmo = (unsigned*)(P.ws + OFF_TMO);
  const unsigned ticket = OFF_TICKET + (par * 8u + (unsigned)dd) * 128u;
  const unsigned want = call & 0xffffffu;
  const unsigned dep4 = (unsigned)(lane & 3) * 4u;
  if (sw == 0) __hip_atomic_store((unsigned*)(P.ws + (ticket ^ 1024u)), 0u, __ATOMIC_RELAXED, __HIP_MEMORY_SCOPE_AGENT);
  auto piece = [&](int p) {
    return decode(__builtin_amdgcn_readlane(ent[0], p >> 3), __builtin_amdgcn_readlane(ent[1], p >> 3), (unsigned)(p + rank) & 7u, spar, fpar);
  };

  unsigned prev_flag = 0xffffffffu, prev_tag = 0;
  Desc nn = piece(8 * NOPEN + 8 + sw); int have_nn = 1;
  auto flush_prev = [&]() {
    if (prev_flag != 0xffffffffu) {
      if (lane == 0) __builtin_amdgcn_raw_buffer_store_b32(prev_tag, rd, 0, uni(prev_flag), AUX_SYS);
      prev_flag = 0xffffffffu;
    }
  };
  auto draw = [&]() {
    if (lane == 0)
      asm volatile("v_mov_b32 v253, 1\n\ts_nop 4\n\tbuffer_atomic_add v253, off, %0, %1 sc0" :: "s"(rl), "s"(ticket) : "memory", ACLOB);
  };
  auto look = [&]() {
    const int p = 8 * NOPEN + 16 + (int)uni(a_get<253>());
    have_nn = 0;
    if (p < nitem) {
      nn = piece(p);
      have_nn = 1;
      a_sample<252>(rl, nn.isout ? dep4 : 0u, uni(nn.dep));
      draw();
    }
  };
  auto half = [&](auto MODE, auto FA, const Desc& d, unsigned lo) {
    constexpr int M = decltype(MODE)::value, F0 = decltype(FA)::value;
    const rsrc_t r = d.isout ? ro : rd;
    const unsigned so = uni(d.dst), st = d.isout ? 16384u : 1024u;
    asm volatile("s_nop 4" ::: "memory");
    static_for<F0, F0 + 4>([&](auto FI) {
      constexpr int f = decltype(FI)::value;
      a_store<f>(r, l16, so + f * st);
      if (M == 1) a_load<f>(rl, l16, lo + f * 1024);
      wait_vm<M == 1 ? 2 * D - 1 : D - 1>();
      if (f == D - 2) flush_prev();
    });
  };
  typedef std::integral_constant<int, 0> I0;
  typedef std::integral_constant<int, 1> I1;
  typedef std::integral_constant<int, 4> I4;
  const u64 t0 = now();
  auto fetch = [&](const Desc& d, unsigned& dtag) -> bool {
    while (true) {
      const unsigned fl = __builtin_amdgcn_raw_buffer_load_b32(rl, d.isout ? dep4 : 0u, uni(d.dep), AUX_SYS);
      dtag = uni(fl);
      if (__all((fl >> 8) == want)) break;
      const u64 dt = now() - t0;
      if (dt > 100000ull) {
        const unsigned tv = uni(__hip_atomic_load(tmo, __ATOMIC_RELAXED, __HIP_MEMORY_SCOPE_AGENT));
        if (tv == call || dt > DEADLINE_TICKS) {
          __hip_atomic_store(tmo, call, __ATOMIC_RELAXED, __HIP_MEMORY_SCOPE_AGENT);
          return false;
        }
      }
      asm volatile("" ::: "memory");
      __builtin_amdgcn_s_sleep(1);
    }
    const unsigned lo = uni(d.src);
    asm volatile("s_nop 4" ::: "memory");
    static_for<0, 8>([&](auto FI) { a_load<decltype(FI)::value>(rl, l16, lo + decltype(FI)::value * 1024); });
    if (have_nn) a_sample<252>(rl, nn.isout ? dep4 : 0u, uni(nn.dep));
    wait_vm<0>();
    return true;
  };
  Desc cur = piece(8 * NOPEN + sw);
  unsigned tag = 0;
  draw();
  if (!fetch(cur, tag)) return;
  STAMP(1);
  while (true) {
    const unsigned fl0 = a_get<252>();
    const Desc nxt = nn; const int have_nxt = have_nn;
    unsigned ntag = uni(fl0);
    const unsigned lo = uni(nxt.src);
    const bool ready = have_nxt && __all((fl0 >> 8) == want);
    if (have_nxt) look();
    if (ready) {
      half(I1{}, I0{}, cur, lo);
      half(I1{}, I4{}, cur, lo);
      prev_flag = cur.flag; prev_tag = tag;
    } else {
      bool late = false;
      if (have_nxt) a_sample<254>(rl, nxt.isout ? dep4 : 0u, uni(nxt.dep));
      half(I0{}, I0{}, cur, 0);
      if (have_nxt) {
        wait_vm<4>();
        const unsigned fl1 = a_get<254>();
        ntag = uni(fl1);
        late = __all((fl1 >> 8) == want);
      }
      if (late) {
        asm volatile("s_nop 4" ::: "memory");
        static_for<0, 4>([&](auto FI) { a_load<decltype(FI)::value>(rl, l16, lo + decltype(FI)::value * 1024); });
        half(I1{}, I4{}, cur, lo);
        wait_vm<8>();
        flush_prev();
        prev_flag = cur.flag; prev_tag = tag;
      } else {
        half(I0{}, I4{}, cur, 0);
        wait_vm<0>();
        flush_prev();
        prev_flag = cur.flag; prev_tag = tag;
        flush_prev();
        if (!have_nxt || !fetch(nxt, ntag)) break;
      }
    }
    cur = nxt; tag = ntag;
  }
  STAMP(11);
}

typedef float f2 __attribute__((ext_vector_type(2)));
__device__ __forceinline__ f2 gelu2(f2 y) {
  const float c1 = -2.0f * 0.7978845608028654f * 1.4426950408889634f;
  const float c3 = c1 * 0.044715f;
  const f2 z = y * __builtin_elementwise_fma(y * y, f2{c3, c3}, f2{c1, c1});
  const f2 e = f2{__builtin_amdgcn_exp2f(z[0]), __builtin_amdgcn_exp2f(z[1])} + f2{1.0f, 1.0f};
  return y * f2{__builtin_amdgcn_rcpf(e[0]), __builtin_amdgcn_rcpf(e[1])};
}

__device__ __forceinline__ void dma8(unsigned m0v, unsigned voff, rsrc_t rs, unsigned soff) {
  asm volatile("s_mov_b32 m0, %0\n\ts_nop 3\n\t"
               "buffer_load_dwordx4 %1, %2, %3 offen " DMA_POL " lds\n\t"
               "buffer_load_dwordx4 %1, %2, %3 offen offset:1024 " DMA_POL " lds\n\t"
               "buffer_load_dwordx4 %1, %2, %3 offen offset:2048 " DMA_POL " lds\n\t"
               "buffer_load_dwordx4 %1, %2, %3 offen offset:3072 " DMA_POL " lds\n\t"
               "s_mov_b32 m0, %4\n\ts_nop 0\n\t"
               "buffer_load_dwordx4 %1, %2, %5 offen " DMA_POL " lds\n\t"
               "buffer_load_dwordx4 %1, %2, %5 offen offset:1024 " DMA_POL " lds\n\t"
               "buffer_load_dwordx4 %1, %2, %5 offen offset:2048 " DMA_POL " lds\n\t"
               "buffer_load_dwordx4 %1, %2, %5 offen offset:3072 " DMA_POL " lds"
               :: "s"(m0v), "v"(voff), "s"(rs), "s"(soff), "s"(m0v + 4096u), "s"(soff + 4096u) : "memory");
}
__device__ __forceinline__ void dma_poll2(unsigned m0v, unsigned voff, rsrc_t rs, unsigned soff) {
  asm volatile("s_mov_b32 m0, %0\n\ts_nop 3\n\t"
               "buffer_load_dword %1, %2, %3 offen sc0 sc1 lds\n\t"
               "buffer_load_dword %1, %2, %3 offen offset:256 sc0 sc1 lds"
               :: "s"(m0v), "v"(voff), "s"(rs), "s"(soff) : "memory");
}
__device__ __forceinline__ void dma_poll1(unsigned m0v, unsigned voff, rsrc_t rs, unsigned soff) {
  asm volatile("s_mov_b32 m0, %0\n\ts_nop 3\n\t"
               "buffer_load_dword %1, %2, %3 offen sc0 sc1 lds"
               :: "s"(m0v), "v"(voff), "s"(rs), "s"(soff) : "memory");
}

#define BARRIER() asm volatile("s_waitcnt lgkmcnt(0)\n\ts_barrier" ::: "memory")

#define SYNC() do { __builtin_amdgcn_sched_barrier(0); __builtin_amdgcn_s_waitcnt(0xC07F); BARRIER(); __builtin_amdgcn_sched_barrier(0); } while (0)

__device__ __forceinline__ void gemm_tile(const Params& P, int rank, lds_ptr smem, int a, int cb0, bool remote,
                                          unsigned call, int wv, int lane) {
  constexpr int STAGE_B = 3 * PIECE;
  constexpr int NS = 6;
  constexpr int MF = 2;
  const int xb0 = (remote ? 8 : 0) + 2 * a;
  const unsigned lds0 = (unsigned)(uintptr_t)smem;
  volatile LAS unsigned* snap = (volatile LAS unsigned*)(smem + LDS_SNAP);
  volatile LAS unsigned* queue = (volatile LAS unsigned*)(smem + LDS_QUEUE);
  volatile LAS unsigned* bad = (volatile LAS unsigned*)(smem + LDS_BAD);
  WAIT_VM0();
  if (wv == 7) { snap[lane] = 0; snap[64 + lane] = 0; snap[128 + lane] = 0; queue[lane] = 0; *bad = 0; }
  BARRIER();
  const unsigned par = call & 1u;
  const unsigned stage = OFF_STAGE + par * STAGE_PAR, flagb = OFF_FLAG + par * FLAG_PAR;
  const rsrc_t rs = mk_rsrc(P.ws, WS_USED);
  STAMP(1);

  if (wv == 7) {
    const unsigned l4 = (unsigned)lane * 4u;
    const unsigned want = call & 0xffffffu;
    unsigned* tmo = (unsigned*)(P.ws + OFF_TMO);
    const unsigned my_entry = 1u | stage_off(0, lane >> 3, lane & 7);
    u64 todo = ~0ull;
    int qn = 0;
    auto poll = [&]() {
      if ((todo >> lane) & 1ull) {
        dma_poll2(lds0 + LDS_SNAP, l4, rs, flagb + (unsigned)xb0 * 256u);
        dma_poll1(lds0 + LDS_SNAP + 512, l4, rs, flagb + (unsigned)(16 + cb0) * 256u);
      }
    };
    auto push = [&](u64 m) {
      const int pos = qn + __builtin_amdgcn_mbcnt_hi((unsigned)(m >> 32), __builtin_amdgcn_mbcnt_lo((unsigned)m, 0u));
      if ((m >> lane) & 1ull) queue[pos] = my_entry;
      todo &= ~m;
      qn += __builtin_popcountll(m);
    };
    auto plan = [&]() {
      const unsigned f0 = snap[lane], f1 = snap[64 + lane], f2 = snap[128 + lane];
      const u64 m = __ballot((f0 >> 8) == want && (f1 >> 8) == want && (f2 >> 8) == want) & todo;
      if (m) push(m);
    };
    for (int d = 0; d < NSTEP; ++d) {
      if (qn <= d) {
        const u64 t0 = now();
        while (true) {
          poll();
          wait_vm<POLL_AHEAD * 3>();
          plan();
          if (qn > d) break;
          const u64 dt = now() - t0;
          if (dt > 100000ull) {
            const unsigned tv = uni(__hip_atomic_load(tmo, __ATOMIC_RELAXED, __HIP_MEMORY_SCOPE_AGENT));
            if (tv == call || dt > DEADLINE_TICKS) {
              __hip_atomic_store(tmo, call, __ATOMIC_RELAXED, __HIP_MEMORY_SCOPE_AGENT);
              *bad = 1;
              push(todo);
              break;
            }
          }
          __builtin_amdgcn_s_sleep(POLL_SLEEP);
        }
        if (todo) { __builtin_amdgcn_s_sleep(POLL_LATE); poll(); }
      }
      BARRIER();
      if (todo) { plan(); if (todo) poll(); }
    }
    BARRIER();
    return;
  }
  if (wv >= 4) {
    const int pi = wv - 4;
    const int myB = pi < 2 ? xb0 + pi : 16 + cb0;
    const unsigned l16 = (unsigned)lane * 16u;
    const unsigned src0 = stage + (unsigned)myB * PIECE - 1u;
    const unsigned dst0 = lds0 + (unsigned)pi * PIECE, dst_end = dst0 + NS * STAGE_B;
    unsigned dst = dst0;
    int issued = 0, avail = 0;
    unsigned qv = 0;
    auto refresh = [&]() {
      qv = queue[lane];
      const u64 m = ~__ballot(qv != 0u);
      avail = m == 0ull ? 64 : __builtin_ctzll(m);
    };
    auto issue1 = [&]() {
      dma8(dst, l16, rs, src0 + __builtin_amdgcn_readlane(qv, issued));
      ++issued;
      dst += STAGE_B;
      if (dst == dst_end) dst = dst0;
    };
    for (int d = 0; d < NSTEP; ++d) {
      if (issued <= d) {
        while (true) { refresh(); if (avail > d) break; __builtin_amdgcn_s_sleep(1); }
        const int upto = avail < d + NS - 1 ? avail : d + NS - 1;
        while (issued < upto) issue1();
      }
      const int n = issued - d;
      wait_vm<32>();
      if (n < 5) { wait_vm<24>(); if (n < 4) { wait_vm<16>(); if (n < 3) { wait_vm<8>(); if (n < 2) wait_vm<0>(); } } }
      BARRIER();
      if (avail < d + NS && avail < NSTEP) refresh();
      const int upto = avail < d + NS ? avail : d + NS;
      while (issued < upto) issue1();
    }
    BARRIER();
    return;
  }

  const lds_ptr a_rd = smem + wv * 4096 + lane * 16;
  const lds_ptr b_rd = smem + 2 * PIECE + lane * 16;
  f4 acc[MF][4];
#pragma unroll
  for (int m = 0; m < MF; ++m)
#pragma unroll
    for (int i = 0; i < 4; ++i) acc[m][i] = f4{0.f, 0.f, 0.f, 0.f};
  int so = 0;
  auto mma = [&](const h8 (&af)[MF][2], const h8 (&bf)[2][4]) {
#pragma unroll
    for (int kb = 0; kb < 2; ++kb)
#pragma unroll
      for (int m = 0; m < MF; ++m)
#pragma unroll
        for (int i = 0; i < 4; ++i)
          acc[m][i] = __builtin_amdgcn_mfma_f32_16x16x32_f16(af[m][kb], bf[kb][i], acc[m][i], 0, 0, 0);
  };
  auto rd = [&](h8 (&an)[MF][2], h8 (&bn)[2][4]) {
#pragma unroll
    for (int kb = 0; kb < 2; ++kb) {
#pragma unroll
      for (int m = 0; m < MF; ++m) an[m][kb] = *(const LAS h8*)(a_rd + so + m * 2048 + kb * 1024);
#pragma unroll
      for (int i = 0; i < 4; ++i) bn[kb][i] = *(const LAS h8*)(b_rd + so + (kb * 4 + i) * 1024);
    }
    so = (so + STAGE_B == NS * STAGE_B) ? 0 : so + STAGE_B;
  };
  auto mix = [&]() {
#pragma unroll
    for (int i = 0; i < 2 * MF + 8; ++i) {
      __builtin_amdgcn_sched_group_barrier(0x100, 1, 0);
      __builtin_amdgcn_sched_group_barrier(0x8, 1, 0);
    }
    __builtin_amdgcn_sched_group_barrier(0x8, 8 * MF - (2 * MF + 8), 0);
  };
  const unsigned row = (unsigned)(wv * 32 + (lane >> 4) * 4);
  const rsrc_t ro = remote ? rs : mk_rsrc(P.out, SHARD_BYTES);
  const unsigned ooff = remote ? OFF_BOX + (unsigned)(1024 + a * 128 + (row >> 3) * 8 + (cb0 >> 2)) * PIECE + (row & 7) * 1024u +
                                     (unsigned)((cb0 & 3) * 256 + (lane & 15) * 16)
                               : ((unsigned)(a * 128) + row) * 16384u + (unsigned)((rank >> 2) * 2048 + cb0 * 64 + (lane & 15) * 4) * 4u;
  const unsigned rstride = remote ? 1024u : 16384u, mstride = remote ? 16u * PIECE : 16u * 16384u;
  h8 a0[MF][2], b0[2][4], a1[MF][2], b1[2][4];
  BARRIER();
  rd(a0, b0);
  for (int d = 1; d < NSTEP - 1; d += 2) {
    SYNC();
    rd(a1, b1); mma(a0, b0); mix();
    SYNC();
    rd(a0, b0); mma(a1, b1); mix();
  }
  mma(a0, b0);
  SYNC();
  STAMP(9);
  rd(a1, b1);
  const unsigned isbad = *bad;
  mma(a1, b1);

  if (uni(isbad) != 0u) {
    const float qn_ = __builtin_nanf("");
#pragma unroll
    for (int m = 0; m < MF; ++m)
#pragma unroll
      for (int i = 0; i < 4; ++i) acc[m][i] = f4{qn_, qn_, qn_, qn_};
  }
#pragma unroll
  for (int m = 0; m < MF; ++m)
#pragma unroll
    for (int r = 0; r < 4; ++r) {
      const f2 lo = gelu2(f2{acc[m][0][r], acc[m][1][r]}), hi = gelu2(f2{acc[m][2][r], acc[m][3][r]});
      const f4 v = {lo[0], lo[1], hi[0], hi[1]};
      __builtin_amdgcn_raw_buffer_store_b128(__builtin_bit_cast(u4, v), ro, ooff + (unsigned)m * mstride + (unsigned)r * rstride, 0, AUX_DEV);
    }
  STAMP(10);
  if (remote) WAIT_VM0();
  BARRIER();
  if (remote && threadIdx.x == 0)
    __builtin_amdgcn_raw_buffer_store_b32(call << 8, rs, 0, OFF_BOXFLAG + (unsigned)(1024 + a * 32 + cb0) * 4u, AUX_DEV);
}

__device__ __forceinline__ void final_wait(const Params& P, int rank, int wv, int lane) {
  const unsigned call = get_call(P, rank, wv);
  const unsigned flagb = OFF_FLAG + (call & 1u) * FLAG_PAR;
  const rsrc_t rl = mk_rsrc(P.ws, WS_USED);
  unsigned* tmo = (unsigned*)(P.ws + OFF_TMO);
  const unsigned want = call << 8;
  const u64 t0 = now();
  while (true) {
    bool ok = true;
#pragma unroll
    for (int b = 0; b < 2; ++b) {
      const u4 v = __builtin_amdgcn_raw_buffer_load_b128(rl, (unsigned)lane * 16u, flagb + FLAG_OUT + b * 1024, AUX_SYS);
      ok &= v[0] == want && v[1] == want && v[2] == want && v[3] == want;
    }
    if (__all(ok)) break;
    const u64 dt = now() - t0;
    if (dt > 100000ull) {
      const unsigned tv = __builtin_amdgcn_readfirstlane(__hip_atomic_load(tmo, __ATOMIC_RELAXED, __HIP_MEMORY_SCOPE_AGENT));
      if (tv == call || dt > DEADLINE_TICKS) {
        __hip_atomic_store(tmo, call, __ATOMIC_RELAXED, __HIP_MEMORY_SCOPE_AGENT);
        for (int i = lane; i < 512; i += 64)
          P.out[(size_t)((i >> 3) * 8) * 4096 + ((rank >> 2) ^ 1) * 2048 + (i & 7) * 256] = __builtin_nanf("");
        break;
      }
    }
    asm volatile("" ::: "memory");
    __builtin_amdgcn_s_sleep(2);
  }
  STAMP(11);
}

}

__global__ void __launch_bounds__(512, 2) fused_gemm_rs(Params P) {
  __shared__ __attribute__((aligned(1024))) char smem_arr[LDS_TOTAL];
  const lds_ptr smem = (lds_ptr)smem_arr;
  const int wv = __builtin_amdgcn_readfirstlane(threadIdx.x >> 6);
  const int lane = threadIdx.x & 63;
  const int xcd = blockIdx.x & 7, slot = blockIdx.x >> 3;
  const int op = (wv - 5) * NGEMM + ((int)blockIdx.x - SLOT_REM * 8);
  const bool opens = slot >= SLOT_REM && wv >= 5 && op < 56 * NOPEN;
  const int op_link = opens ? (op >> 3) % 7 : 0, op_oct = opens ? (op >> 3) / 7 : 0;
  asm volatile("" :: "s"(P.x), "s"(P.w), "s"(P.ws), "s"(P.rank), "s"(P.link[op_link].list[op_oct]), "s"(P.link[op_link].ws));
  const int rank = P.rank & 7;
  STAMP(0);
  unsigned call;
  if (__builtin_expect(opens, 1)) {
    call = starter(P, rank, op_link + 1, op_oct * 8 + xcd, wv, lane);
  } else if (slot < SLOT_REM) {
    if (wv < 2) {
      const int dd = 1 + slot * 2 + wv;
      if (dd < 8) pusher(P, rank, dd, xcd, wv, lane);
      else if (blockIdx.x == 24) final_wait(P, rank, wv, lane);
    }
    return;
  } else {
    const int box = wv * NGEMM + ((int)blockIdx.x - SLOT_REM * 8);
    if (box < 1024) {
      const unsigned delay = (unsigned)((CONV_DELAYS >> ((box >> 7) * 8)) & 0xffull) * 25u;
      const u64 ts = now();
      while (now() - ts < delay) __builtin_amdgcn_s_sleep(4);
      call = convert(P, rank, box, wv, lane);
    }
    else call = get_call(P, rank, wv);
  }
  STAMP(12);
  const bool first_q = (xcd >> 1) == (rank & 3);
  bool remote = slot < SLOT_OWN;
  int a, cb;
  if (remote) { const int s = slot - SLOT_REM; a = s >> 2; cb = xcd * 4 + (s & 3); }
  else if (!first_q) { const int s = slot - SLOT_OWN; a = s >> 2; cb = xcd * 4 + (s & 3); }
  else { a = 3; cb = ((rank & 3) * 8 + 8 + (xcd & 1) * 12 + (slot - SLOT_OWN)) & 31; }
  const int ntask = (remote && first_q) ? 2 : 1;
  for (int task = 0; task < ntask; ++task) {
    gemm_tile(P, rank, smem, a, cb, remote, call, wv, lane);
    remote = false;
  }
}

static const char* const kOrder[2] = {
    "H00 W00 H10 W01 W10 W02 H20 W03 W20 F60 W11 H30 F61 W30 W12 F62 H40 W21 W40 F63 W13 H50 F70 W31 W50 F71 H60 W22 "
    "W41 F72 W60 H70 F73 W32 W51 W70 W61 W42 W23 W71 W52 W33 W62 W72 W43 W53 W63 W73 O00 O10 O20 O30 O40 O50 O60 O70",
    "R60 R70 H00 W00 W01 W02 W03 H10 W10 W11 W12 W13 H60 H20 W20 W21 W22 W23 H70 H30 W30 W31 W32 W33 O00 H40 W40 W41 "
    "W42 W43 O10 H50 W50 W51 W52 W53 O20 O30 O40 O50 O60 O70 "
    "Y00 Y10 Y20 Y30 Y01 Y11 Y21 Y31 Y02 Y12 Y22 Y32 Y03 Y13 Y23 Y33"};

static void build_lists(Params& P, int rank) {
  for (int dd = 1; dd < NRANK; ++dd) {
    const int cls = dd >> 2, dest = rank ^ dd, G = dest >> 2, q = dest & 3;
    const char* s = kOrder[cls];
    int n = 0;
    while (*s && n < NOCT) {
      if (*s == ' ') { ++s; continue; }
      const char c = s[0];
      const int t = s[1] - '0', sub = s[2] - '0';
      s += 3;
      unsigned src, kind = 0, dst, flag, dep;
      auto staged = [&](int B, int from) {
        dst = (OFF_STAGE + (unsigned)((from * 8 + t) * 48 + B) * PIECE) / 1024u;
        flag = (unsigned)(B * 64 + t * 8 + from);
      };
      auto boxed = [&](int blk) { src = OFF_BOX / PIECE + (unsigned)(t * 128 + blk); dep = OFF_BOXFLAG / 4u + (unsigned)(t * 128 + blk); };
      if (c == 'W' || c == 'R') {
        const int cb = ((sub + q) & 3) * 8;
        boxed(G * 32 + cb); staged(16 + cb, rank);
      } else if (c == 'H') {
        boxed(64 + 32 * (G ^ 1) + 2 * q); staged(8, rank); kind = 1;
      } else if (c == 'O') {
        boxed(64 + 8 * dest); staged(0, rank);
      } else if (c == 'F') {
        staged(16 + (rank & 3) * 8, (G ^ 1) * 4 + sub);
        src = dst / 8u; dep = OFF_FLAG / 4u + flag; kind = 4;
      } else {
        const int rg = t * 4, cg = ((sub + rank) & 3) * 2;
        src = OFF_BOX / PIECE + (unsigned)(1024 + q * 128 + rg * 8 + cg);
        dst = (unsigned)((rank & 3) * 128 + rg * 8) * 16u + (unsigned)((rank >> 2) * 8 + cg);
        flag = FLAG_OUT / 4u + (unsigned)((rank & 3) * 128 + rg * 8 + cg);
        dep = OFF_BOXFLAG / 4u + 1024u + (unsigned)(q * 32 + cg * 4);
        kind = 3;
      }
      P.link[dd - 1].list[n][0] = src | kind << 13 | dst << 16;
      P.link[dd - 1].list[n][1] = flag | dep << 16;
      ++n;
    }
    P.nitem[cls] = n * 8;
    for (; n < NOCT; ++n) P.link[dd - 1].list[n] = P.link[dd - 1].list[0];
  }
}

extern "C" void kernel_launch(int rank, int nranks, ncclComm_t comm,
                              hipStream_t stream,
                              void* const* d_in, const int* in_sizes, int n_in,
                              void* d_out, int out_size,
                              const int* in_dtypes, int out_dtype,
                              void* const* d_out_all,
                              void* const* d_ws_all, size_t ws_bytes) {
  (void)comm; (void)in_dtypes; (void)out_dtype;
  if (nranks != NRANK || n_in != 2 || in_sizes[0] != 4096 * 512 || in_sizes[1] != 512 * 4096 ||
      out_size != 512 * 4096 || ws_bytes < WS_USED || (unsigned)rank >= (unsigned)NRANK) {
    fprintf(stderr, "kernel_launch: unexpected problem shape\n");
    return;
  }
  Params P;
  P.x = (const float*)d_in[0];
  P.w = (const float*)d_in[1];
  P.ws = (char*)d_ws_all[rank];
  P.out = (float*)d_out;
  P.rank = rank; P.pad = 0;
  for (int dd = 1; dd < NRANK; ++dd) { P.link[dd - 1].ws = (char*)d_ws_all[rank ^ dd]; P.link[dd - 1].out = (float*)d_out_all[rank ^ dd]; }
  build_lists(P, rank);
  fused_gemm_rs<<<256, 512, 0, stream>>>(P);
  HIP_CHECK(hipGetLastError());
}
```

```cpp
#include <hip/hip_runtime.h>
#include <rccl/rccl.h>
#include <cstdio>
#include <cstdint>
#include <cstdlib>
#include <type_traits>

#define HIP_CHECK(expr) do { hipError_t _e = (expr); if (_e != hipSuccess) { \
    fprintf(stderr, "HIP error %s at %s:%d\n", hipGetErrorString(_e), __FILE__, __LINE__); \
    return; } } while (0)

typedef _Float16 h8 __attribute__((ext_vector_type(8)));
typedef float f4 __attribute__((ext_vector_type(4)));
typedef unsigned u4 __attribute__((ext_vector_type(4)));
typedef unsigned u2 __attribute__((ext_vector_type(2)));
typedef __amdgpu_buffer_rsrc_t rsrc_t;
typedef unsigned long long u64;
#define LAS __attribute__((address_space(3)))
typedef LAS char* lds_ptr;

namespace {

constexpr int NRANK = 8;
constexpr int NSTEP = 64;
constexpr unsigned PIECE = 8192;
constexpr unsigned SHARD_BYTES = 8u << 20;
constexpr unsigned OFF_CNT = 0;
constexpr unsigned OFF_TMO = 16384;
constexpr unsigned OFF_TICKET = 20480;
constexpr unsigned OFF_BOXFLAG = 32768;
constexpr unsigned OFF_FLAG = 65536;
constexpr unsigned FLAG_PAR = 65536;
constexpr unsigned FLAG_OUT = 16384;
constexpr unsigned OFF_STAGE = 1u << 20;
constexpr unsigned STAGE_PAR = 48u * 64u * PIECE;
constexpr unsigned OFF_BOX = OFF_STAGE + 2u * STAGE_PAR;
constexpr unsigned WS_USED = OFF_BOX + (1024u + 512u) * PIECE;
constexpr u64 DEADLINE_TICKS = 40000000ull;
constexpr int RS3 = 0x00020000;
constexpr int AUX_SYS = 17;
constexpr int AUX_DEV = 16;
constexpr int NOCT = 60;
#define DMA_POL "sc0 sc1"
constexpr int FINAL_SLEEP = 0;
constexpr int POLL_AHEAD = 1;
constexpr int POLL_LATE = 8;
constexpr int POLL_SLEEP = 2;
constexpr int PUSH_DEPTH = 6;
constexpr int PUSH_GO = 525;
constexpr int NOPEN = 4;
constexpr int OPEN_DEPTH = 2;
constexpr u64 CONV_DELAYS = 0x1E141C1810080202ull;
constexpr int SLOT_REM = 4, SLOT_OWN = SLOT_REM + 16;
constexpr int NGEMM = 256 - SLOT_REM * 8;

constexpr int LDS_RING = 6 * 3 * PIECE;
constexpr int LDS_SNAP = LDS_RING;
constexpr int LDS_QUEUE = LDS_SNAP + 1024;
constexpr int LDS_EXPS = LDS_QUEUE + 256;
constexpr int LDS_BAD = LDS_EXPS + 256;
constexpr int LDS_TOTAL = LDS_BAD + 16;

struct Link {
  char* ws;
  float* out;
  u2 list[NOCT];
};
struct Params {
  const float* x;
  const float* w;
  char* ws;
  float* out;
  int rank;
  int nitem[7];
  Link link[7];
};

__device__ __forceinline__ rsrc_t mk_rsrc(const void* p, unsigned bytes) {
  const u64 a = (u64)p;
  const u64 u = (u64)(unsigned)__builtin_amdgcn_readfirstlane((unsigned)a) |
                ((u64)(unsigned)__builtin_amdgcn_readfirstlane((unsigned)(a >> 32)) << 32);
  return __builtin_amdgcn_make_buffer_rsrc((void*)u, (short)0, (int)bytes, RS3);
}
__device__ __forceinline__ u64 now() { return __builtin_amdgcn_s_memrealtime(); }
#define WAIT_VM0() do { __builtin_amdgcn_s_waitcnt(0x0F70); asm volatile("s_waitcnt vmcnt(0)" ::: "memory"); } while (0)
#define WAIT_LGKM0() asm volatile("s_waitcnt lgkmcnt(0)" ::: "memory")

__device__ __forceinline__ unsigned pk2(float a, float b) {
  typedef _Float16 h2 __attribute__((ext_vector_type(2)));
  h2 v = {(_Float16)a, (_Float16)b};
  return __builtin_bit_cast(unsigned, v);
}

__device__ __forceinline__ float asf(unsigned v) { return __builtin_bit_cast(float, v); }

__device__ __forceinline__ unsigned get_call(const Params& P, int rank, int wv) {
  unsigned* c = (unsigned*)(P.ws + OFF_CNT) + (blockIdx.x * 8 + wv);
  unsigned v = __hip_atomic_load(c, __ATOMIC_RELAXED, __HIP_MEMORY_SCOPE_AGENT) + 1u;
  v = __builtin_amdgcn_readfirstlane(v);
  __hip_atomic_store(c, v, __ATOMIC_RELAXED, __HIP_MEMORY_SCOPE_AGENT);
  return v;
}

template <int AUX>
__device__ __forceinline__ void store_piece(const u4 (&h)[8], rsrc_t rd, unsigned l16, unsigned so) {
#pragma unroll
  for (int f = 0; f < 8; ++f)
    __builtin_amdgcn_raw_buffer_store_b128(h[f], rd, l16 + (f & 3) * 1024, so + (f >> 2) * 4096, AUX);
  asm volatile("s_nop 1" ::: "memory");
}

__device__ __forceinline__ void load_raw(const Params& P, int box, int lane, u4 (&raw)[16]) {
  const unsigned t = (unsigned)box >> 7, RB = (unsigned)box & 63u;
  const bool isw = (box & 64) == 0;
  const rsrc_t rs = mk_rsrc(isw ? (const void*)P.w : (const void*)P.x, SHARD_BYTES);
  const unsigned voff = ((unsigned)(lane >> 4) << (isw ? 16 : 4)) | ((unsigned)(lane & 15) << (isw ? 4 : 11));
  const unsigned src = (t << (isw ? 20 : 8)) | (RB << (isw ? 8 : 17));
  const unsigned s1 = isw ? 262144u : 32768u, s0 = isw ? 16384u : 64u;
#pragma unroll
  for (int n = 0; n < 16; ++n) {
    raw[n] = __builtin_amdgcn_raw_buffer_load_b128(rs, voff, src + (n >> 2) * s1 + (n & 3) * s0, 0);
    if (n == 0) __builtin_amdgcn_sched_barrier(0);
  }
}

__device__ __forceinline__ unsigned to_half(int box, u4 (&raw)[16], u4 (&h)[8]) {
  unsigned m = 0;
#pragma unroll
  for (int n = 0; n < 16; ++n) {
    asm("v_max3_f32 %0, %0, |%1|, |%2|" : "+v"(m) : "v"(raw[n][0]), "v"(raw[n][1]));
    asm("v_max3_f32 %0, %0, |%1|, |%2|" : "+v"(m) : "v"(raw[n][2]), "v"(raw[n][3]));
  }
  int e = 0;
  const u64 too_big = __ballot(m >= ((127u + 15u) << 23)), big_enough = __ballot(m >= ((127u - 8u) << 23));
  if (__builtin_expect_with_probability((too_big != 0ull) | (big_enough == 0ull), 0, 1.0)) {
    m = 0;
#pragma unroll
    for (int n = 0; n < 16; ++n)
#pragma unroll
      for (int i = 0; i < 4; ++i) {
        const unsigned a = raw[n][i] & 0x7fffffffu;
        m = (a < 0x7f800000u && a > m) ? a : m;
      }
#pragma unroll
    for (int o = 32; o > 0; o >>= 1) { const unsigned t = __shfl_xor(m, o); m = m > t ? m : t; }
    m = __builtin_amdgcn_readfirstlane(m);
    if (m != 0u) {
      e = (int)(m >> 23) - 127 - 8;
      e = e < -120 ? -120 : e;
#pragma unroll
      for (int n = 0; n < 16; ++n)
#pragma unroll
        for (int i = 0; i < 4; ++i) raw[n][i] = __builtin_bit_cast(unsigned, __builtin_amdgcn_ldexpf(asf(raw[n][i]), -e));
    }
  }
  if ((box & 64) == 0) {
#pragma unroll
    for (int kb = 0; kb < 2; ++kb)
#pragma unroll
      for (int i = 0; i < 4; ++i)
#pragma unroll
        for (int c = 0; c < 4; ++c)
          h[kb * 4 + i][c] = pk2(asf(raw[kb * 8 + 2 * c][i]), asf(raw[kb * 8 + 2 * c + 1][i]));
  } else {
#pragma unroll
    for (int f = 0; f < 8; ++f) {
      h[f][0] = pk2(asf(raw[2 * f][0]), asf(raw[2 * f][1]));
      h[f][1] = pk2(asf(raw[2 * f][2]), asf(raw[2 * f][3]));
      h[f][2] = pk2(asf(raw[2 * f + 1][0]), asf(raw[2 * f + 1][1]));
      h[f][3] = pk2(asf(raw[2 * f + 1][2]), asf(raw[2 * f + 1][3]));
    }
  }
  return (unsigned)e & 0xffu;
}

__device__ __forceinline__ unsigned stage_off(int B, int t, int src) { return (unsigned)((src * 8 + t) * 48 + B) * PIECE; }
__device__ __forceinline__ unsigned flag_off(int B, int t, int src) { return (unsigned)(B * 64 + t * 8 + src) * 4u; }

struct Desc {
  unsigned src;
  unsigned dst;
  unsigned flag;
  unsigned dep;
  int isout;
};

__device__ __forceinline__ Desc decode(unsigned lo, unsigned hi, unsigned v, unsigned spar, unsigned fpar) {
  const unsigned kind = (lo >> 13) & 7u;
  const bool isout = kind & 2u, fwd = kind == 4u, two = kind == 5u;
  const unsigned w8 = (v >> 1) * 8u + (v & 1u), c = v & 3u, r = v >> 2;
  const unsigned si = two ? c + r * 128u : kind == 2u ? v * 8u : (kind & 1u) ? w8 : v;
  const unsigned fi = two ? c * 64u + r * 8u : v * 64u;
  const unsigned di = two ? (c + r * 48u) * 8u : kind == 2u ? v * 128u : kind == 3u ? (v >> 1) * 128u + (v & 1u) : v * 8u;
  Desc d;
  d.src = ((lo & 0x1fffu) + si) * PIECE + (fwd ? spar : 0u);
  d.dst = ((lo >> 16) + di) * 1024u + (isout ? 0u : spar);
  d.flag = isout ? 0xffffffffu : ((hi & 0xffffu) + fi) * 4u + OFF_FLAG + fpar;
  d.dep = ((hi >> 16) + (isout ? (v & kind & 1u) * 4u : fwd ? fi : si)) * 4u + (fwd ? fpar : 0u);
  d.isout = isout;
  return d;
}

template <int N> __device__ __forceinline__ void wait_vm() { asm volatile("s_waitcnt vmcnt(%0)" :: "i"(N) : "memory"); }
constexpr int AB = 220;
#define ACLOB "v219","v220","v221","v222","v223","v224","v225","v226","v227","v228","v229","v230","v231","v232","v233","v234","v235","v236","v237","v238","v239","v240","v241","v242","v243","v244","v245","v246","v247","v248","v249","v250","v251","v252","v253","v254","v255"
__device__ __forceinline__ unsigned uni(unsigned v) { return __builtin_amdgcn_readfirstlane(v); }
template <int F> __device__ __forceinline__ void a_store(rsrc_t r, unsigned vo, unsigned so) {
  asm volatile("buffer_store_dwordx4 v[%c3:%c4], %0, %1, %2 offen sc0 sc1" :: "v"(vo), "s"(r), "s"(so), "i"(AB + F * 4), "i"(AB + F * 4 + 3) : "memory", ACLOB);
}
template <int F> __device__ __forceinline__ void a_load(rsrc_t r, unsigned vo, unsigned so) {
  asm volatile("buffer_load_dwordx4 v[%c3:%c4], %0, %1, %2 offen offset:%c5 sc0 sc1"
               :: "v"(vo), "s"(r), "s"(so), "i"(AB + F * 4), "i"(AB + F * 4 + 3), "i"((F & 3) * 1024) : "memory", ACLOB);
}
template <int R> __device__ __forceinline__ void a_sample(rsrc_t r, unsigned vo, unsigned so) {
  asm volatile("buffer_load_dword v[%c3], %0, %1, %2 offen sc0 sc1" :: "v"(vo), "s"(r), "s"(so), "i"(R) : "memory", ACLOB);
}
template <int R> __device__ __forceinline__ unsigned a_get() {
  unsigned v;
  asm volatile("v_mov_b32 %0, v[%c1]\n\ts_nop 1" : "=v"(v) : "i"(R) : "memory", ACLOB);
  return v;
}
template <int I, int N, class Fn> __device__ __forceinline__ void static_for(Fn&& fn) {
  if constexpr (I < N) { fn(std::integral_constant<int, I>{}); static_for<I + 1, N>(fn); }
}

__device__ __forceinline__ unsigned convert(const Params& P, int rank, bool opens, const Link& L, u2 e, unsigned v, int box,
                                            int wv, int lane) {
  if (opens) box = (int)((e[0] & 0x1fffu) + ((e[0] & 0x2000u) ? (v >> 1) * 8u + (v & 1u) : v)) - (int)(OFF_BOX / PIECE);
  u4 raw[16], h[8];
  unsigned* cnt_p = (unsigned*)(P.ws + OFF_CNT) + (blockIdx.x * 8 + wv);
  load_raw(P, box, lane, raw);
  const unsigned cnt_v = __hip_atomic_load(cnt_p, __ATOMIC_RELAXED, __HIP_MEMORY_SCOPE_AGENT);
  const rsrc_t rd = mk_rsrc(opens ? L.ws : P.ws, WS_USED);
  const unsigned touch = __builtin_amdgcn_raw_buffer_load_b32(rd, opens ? ((e[0] >> 16) + v * 8u) * 1024u : 0xfffffff0u, 0, AUX_SYS);
  const unsigned ex = to_half(box, raw, h);
  const unsigned call = uni(cnt_v) + 1u;
  const unsigned spar = (call & 1u) * STAGE_PAR, fpar = (call & 1u) * FLAG_PAR;
  const unsigned l16 = (unsigned)lane * 16u;
  unsigned dst, flag, dst2 = 0, flag2 = 0;
  if (opens) {
    dst = ((e[0] >> 16) + v * 8u) * 1024u + spar;
    flag = ((e[1] & 0xffffu) + v * 64u) * 4u + OFF_FLAG + fpar;
  } else {
    const int t = box >> 7, RB = box & 63;
    const int owner = RB >> 3;
    const int helper = ((owner >> 2) ^ 1) * 4 + ((RB & 7) >> 1);
    int selfB = -1;
    if ((box & 64) == 0) { if ((RB >> 5) == (rank >> 2)) selfB = 16 + (RB & 31); }
    else if (owner == rank) selfB = RB & 7;
    else if (helper == rank) selfB = 8 + 2 * (owner & 3) + (RB & 1);
    dst = OFF_BOX + (unsigned)box * PIECE; flag = OFF_BOXFLAG + (unsigned)box * 4u;
    if (selfB >= 0) { dst2 = OFF_STAGE + spar + stage_off(selfB, t, rank); flag2 = OFF_FLAG + fpar + flag_off(selfB, t, rank); }
  }
  const unsigned tag = call << 8 | ex;
  if (opens) {
#pragma unroll
    for (int f = 0; f < 8; ++f) {
      __builtin_amdgcn_raw_buffer_store_b128(h[f], rd, l16 + f * 1024, uni(dst), AUX_SYS);
      wait_vm<OPEN_DEPTH - 1>();
    }
    asm volatile("s_nop 1" ::: "memory");
  } else {
    store_piece<AUX_SYS>(h, rd, l16, uni(dst));
  }
  if (dst2) store_piece<AUX_SYS>(h, rd, l16, uni(dst2));
  __hip_atomic_store(cnt_p, call, __ATOMIC_RELAXED, __HIP_MEMORY_SCOPE_AGENT);
  WAIT_VM0();
  asm volatile("" :: "v"(touch));
  if (lane == 0) {
    __builtin_amdgcn_raw_buffer_store_b32(tag, rd, 0, uni(flag), AUX_SYS);
    if (dst2) __builtin_amdgcn_raw_buffer_store_b32(tag, rd, 0, uni(flag2), AUX_SYS);
  }
  return call;
}

__device__ __forceinline__ void pusher(const Params& P, int rank, lds_ptr smem, int dd, int sw, int wv, int lane) {
  constexpr int D = PUSH_DEPTH;
  static_assert(D >= 3 && D <= 6, "window bookkeeping below");
  constexpr unsigned NONE = 0xfffffff0u;
  const u64 t_in = now();
  const Link& L = P.link[dd - 1];
  const unsigned l16 = (unsigned)lane * 16u;
  const u2 ent = L.list[lane < NOCT ? lane : 0];
  const unsigned call = get_call(P, rank, wv);
  const int nitem = P.nitem[dd - 1];
  const rsrc_t rl = mk_rsrc(P.ws, WS_USED);
  const rsrc_t rd = mk_rsrc(L.ws, WS_USED);
  const rsrc_t ro = mk_rsrc(L.out, SHARD_BYTES);
  const unsigned par = call & 1u;
  const unsigned spar = par * STAGE_PAR, fpar = par * FLAG_PAR;
  unsigned* tmo = (unsigned*)(P.ws + OFF_TMO);
  const unsigned ticket = OFF_TICKET + (par * 8u + (unsigned)dd) * 128u;
  const unsigned want = call & 0xffffffu;
  const unsigned dep4 = (unsigned)(lane & 3) * 4u;
  if (sw == 0) __hip_atomic_store((unsigned*)(P.ws + (ticket ^ 1024u)), 0u, __ATOMIC_RELAXED, __HIP_MEMORY_SCOPE_AGENT);
  LAS u4* tab = (LAS u4*)(smem + wv * 8192);
#pragma unroll
  for (int u = 0; u < 8; ++u) {
    const Desc d = decode(ent[0], ent[1], (unsigned)(u + rank) & 7u, spar, fpar);
    tab[lane * 8 + u] = u4{d.src, d.dst, d.isout ? NONE : d.flag, d.dep};
  }
  WAIT_LGKM0();

  unsigned prev_flag = NONE, prev_tag = 0;
  auto flush_prev = [&]() {
    asm volatile("s_mov_b64 exec, 1\n\tv_mov_b32 v255, %0\n\tv_mov_b32 v219, %2\n\t"
                 "buffer_store_dword v255, v219, %1, 0 offen sc0 sc1\n\ts_mov_b64 exec, -1"
                 :: "s"(prev_tag), "s"(rd), "s"(prev_flag) : "memory", ACLOB);
    prev_flag = NONE;
  };
  auto draw = [&]() {
    asm volatile("s_mov_b64 exec, 1\n\tv_mov_b32 v253, 1\n\tbuffer_atomic_add v253, off, %0, %1 sc0\n\ts_mov_b64 exec, -1"
                 :: "s"(rl), "s"(ticket) : "memory", ACLOB);
  };
  auto dep_of = [&](const u4& q) -> unsigned { return q[3] + (q[2] == NONE ? dep4 : 0u); };
  unsigned cur_dst, cur_flag, cur_tag = 0, nxt_src = 0, nxt_dst = 0, nxt_flag = NONE, nxt_tag = 0, nxt_dep = 0;
  bool have_nxt = false, have_nn = true, ready = false;
  u4 nq = tab[8 * NOPEN + 8 + sw], nq_new = nq;
  unsigned n_src = 0, n_dst = 0, n_flag = 0, n_dep = 0;
  bool have_new = false;
  auto adv_a = [&]() {
    const int p = 8 * NOPEN + 16 + (int)uni(a_get<253>());
    have_new = have_nn && p < nitem;
    nq_new = tab[have_new ? p : 0];
    n_src = uni(nq[0]); n_dst = uni(nq[1]); n_flag = uni(nq[2]); n_dep = dep_of(nq);
  };
  auto adv_b = [&]() {
    const unsigned fl = a_get<252>();
    cur_dst = nxt_dst; cur_flag = nxt_flag; cur_tag = nxt_tag;
    nxt_src = n_src; nxt_dst = n_dst; nxt_flag = n_flag; nxt_dep = n_dep; nxt_tag = uni(fl);
    have_nxt = have_nn;
    ready = have_nxt && __all((fl >> 8) == want);
    have_nn = have_new; nq = nq_new;
    a_sample<252>(rl, dep_of(nq), 0u);
    draw();
  };
  auto half = [&](auto MODE, auto FA, rsrc_t r, unsigned& so, unsigned st, unsigned lo) {
    constexpr int M = decltype(MODE)::value, F0 = decltype(FA)::value;
    static_for<F0, F0 + 4>([&](auto FI) {
      constexpr int f = decltype(FI)::value;
      a_store<f>(r, l16, so);
      if (M) a_load<f>(rl, l16, lo + F0 * 1024);
      so += st;
      asm volatile("" : "+s"(so));
      if (M == 2 && f == 6) adv_a();
      if (f < 7) wait_vm<M ? 2 * D - 1 + (f < D - 1 ? 2 : 0) + ((f >= D - 1 || f + 8 <= 2 * D - 3) ? 1 : 0) : (f < D - 1 ? 2 * D - 2 - f : D - 1)>();
      if (f == D - 2) flush_prev();
    });
  };
  typedef std::integral_constant<int, 0> I0;
  typedef std::integral_constant<int, 1> I1;
  typedef std::integral_constant<int, 2> I2;
  typedef std::integral_constant<int, 4> I4;
  const u64 t0 = now();
  auto fetch = [&]() -> bool {
    while (true) {
      const unsigned fl = __builtin_amdgcn_raw_buffer_load_b32(rl, nxt_dep, 0, AUX_SYS);
      nxt_tag = uni(fl);
      if (__all((fl >> 8) == want)) break;
      const u64 dt = now() - t0;
      if (dt > 100000ull) {
        const unsigned tv = uni(__hip_atomic_load(tmo, __ATOMIC_RELAXED, __HIP_MEMORY_SCOPE_AGENT));
        if (tv == call || dt > DEADLINE_TICKS) {
          __hip_atomic_store(tmo, call, __ATOMIC_RELAXED, __HIP_MEMORY_SCOPE_AGENT);
          return false;
        }
      }
      asm volatile("" ::: "memory");
      __builtin_amdgcn_s_sleep(1);
    }
    static_for<0, 8>([&](auto FI) { a_load<decltype(FI)::value>(rl, l16, nxt_src + (decltype(FI)::value & 4) * 1024); });
    a_sample<252>(rl, dep_of(nq), 0u);
    wait_vm<0>();
    return true;
  };
  draw();
  {
    const u4 q = tab[8 * NOPEN + sw];
    nxt_src = uni(q[0]); nxt_dst = uni(q[1]); nxt_flag = uni(q[2]); nxt_dep = dep_of(q);
    asm volatile("s_nop 4" ::: "memory");
  }
  if (!fetch()) return;
  while (now() - t_in < PUSH_GO) __builtin_amdgcn_s_sleep(1);
  adv_a();
  while (true) {
    adv_b();
    const bool isout = cur_flag == NONE;
    const rsrc_t r = isout ? ro : rd;
    const unsigned st = isout ? 16384u : 1024u;
    unsigned so = cur_dst;
    if (__builtin_expect(ready, 1)) {
      wait_vm<2 * D + 2>();
      half(I1{}, I0{}, r, so, st, nxt_src);
      half(I2{}, I4{}, r, so, st, nxt_src);
      prev_flag = cur_flag; prev_tag = cur_tag;
    } else if (!have_nxt) {
      static_for<0, 8>([&](auto FI) {
        constexpr int f = decltype(FI)::value;
        wait_vm<14 - f>();
        a_store<f>(r, l16, so);
        so += st;
      });
      wait_vm<8>();
      flush_prev();
      wait_vm<1>();
      prev_flag = cur_flag; prev_tag = cur_tag;
      flush_prev();
      if (dd >= 4) { prev_flag = OFF_FLAG + fpar + FLAG_OUT + (unsigned)((rank & 3) * 8 + sw) * 4u; prev_tag = call << 8; flush_prev(); }
      break;
    } else {
      a_sample<254>(rl, nxt_dep, 0u);
      wait_vm<2 * D - 1>();
      half(I0{}, I0{}, r, so, st, 0);
      wait_vm<4>();
      const unsigned fl1 = a_get<254>();
      nxt_tag = uni(fl1);
      if (__all((fl1 >> 8) == want)) {
        static_for<0, 4>([&](auto FI) { a_load<decltype(FI)::value>(rl, l16, nxt_src); });
        half(I1{}, I4{}, r, so, st, nxt_src);
        wait_vm<8>();
        flush_prev();
        prev_flag = cur_flag; prev_tag = cur_tag;
      } else {
        half(I0{}, I4{}, r, so, st, 0);
        wait_vm<0>();
        flush_prev();
        prev_flag = cur_flag; prev_tag = cur_tag;
        flush_prev();
        if (!fetch()) break;
      }
      adv_a();
    }
  }
}

typedef float f2 __attribute__((ext_vector_type(2)));
__device__ __forceinline__ f2 gelu2(f2 y) {
  const float c1 = -2.0f * 0.7978845608028654f * 1.4426950408889634f;
  const float c3 = c1 * 0.044715f;
  const f2 z = y * __builtin_elementwise_fma(y * y, f2{c3, c3}, f2{c1, c1});
  const f2 e = f2{__builtin_amdgcn_exp2f(z[0]), __builtin_amdgcn_exp2f(z[1])} + f2{1.0f, 1.0f};
  return y * f2{__builtin_amdgcn_rcpf(e[0]), __builtin_amdgcn_rcpf(e[1])};
}

__device__ __forceinline__ void dma8(unsigned m0v, unsigned voff, rsrc_t rs, unsigned soff) {
  asm volatile("s_mov_b32 m0, %0\n\ts_nop 3\n\t"
               "buffer_load_dwordx4 %1, %2, %3 offen " DMA_POL " lds\n\t"
               "buffer_load_dwordx4 %1, %2, %3 offen offset:1024 " DMA_POL " lds\n\t"
               "buffer_load_dwordx4 %1, %2, %3 offen offset:2048 " DMA_POL " lds\n\t"
               "buffer_load_dwordx4 %1, %2, %3 offen offset:3072 " DMA_POL " lds\n\t"
               "s_mov_b32 m0, %4\n\ts_nop 0\n\t"
               "buffer_load_dwordx4 %1, %2, %5 offen " DMA_POL " lds\n\t"
               "buffer_load_dwordx4 %1, %2, %5 offen offset:1024 " DMA_POL " lds\n\t"
               "buffer_load_dwordx4 %1, %2, %5 offen offset:2048 " DMA_POL " lds\n\t"
               "buffer_load_dwordx4 %1, %2, %5 offen offset:3072 " DMA_POL " lds"
               :: "s"(m0v), "v"(voff), "s"(rs), "s"(soff), "s"(m0v + 4096u), "s"(soff + 4096u) : "memory");
}
__device__ __forceinline__ void dma_poll2(unsigned m0v, unsigned voff, rsrc_t rs, unsigned soff) {
  asm volatile("s_mov_b32 m0, %0\n\ts_nop 3\n\t"
               "buffer_load_dword %1, %2, %3 offen sc0 sc1 lds\n\t"
               "buffer_load_dword %1, %2, %3 offen offset:256 sc0 sc1 lds"
               :: "s"(m0v), "v"(voff), "s"(rs), "s"(soff) : "memory");
}
__device__ __forceinline__ void dma_poll1(unsigned m0v, unsigned voff, rsrc_t rs, unsigned soff) {
  asm volatile("s_mov_b32 m0, %0\n\ts_nop 3\n\t"
               "buffer_load_dword %1, %2, %3 offen sc0 sc1 lds"
               :: "s"(m0v), "v"(voff), "s"(rs), "s"(soff) : "memory");
}

#define BARRIER() asm volatile("s_waitcnt lgkmcnt(0)\n\ts_barrier" ::: "memory")

#define SYNC() do { __builtin_amdgcn_sched_barrier(0); __builtin_amdgcn_s_waitcnt(0xC07F); BARRIER(); __builtin_amdgcn_sched_barrier(0); } while (0)

__device__ __forceinline__ void gemm_tile(const Params& P, int rank, lds_ptr smem, int a, int cb0, bool remote,
                                          unsigned call, int wv, int lane) {
  constexpr int STAGE_B = 3 * PIECE;
  constexpr int NS = 6;
  constexpr int MF = 2;
  const int xb0 = (remote ? 8 : 0) + 2 * a;
  const unsigned lds0 = (unsigned)(uintptr_t)smem;
  volatile LAS unsigned* snap = (volatile LAS unsigned*)(smem + LDS_SNAP);
  volatile LAS unsigned* queue = (volatile LAS unsigned*)(smem + LDS_QUEUE);
  volatile LAS unsigned* exps = (volatile LAS unsigned*)(smem + LDS_EXPS);
  volatile LAS unsigned* bad = (volatile LAS unsigned*)(smem + LDS_BAD);
  WAIT_VM0();
  if (wv == 7) { snap[lane] = 0; snap[64 + lane] = 0; snap[128 + lane] = 0; queue[lane] = 0; *bad = 0; }
  BARRIER();
  const unsigned par = call & 1u;
  const unsigned stage = OFF_STAGE + par * STAGE_PAR, flagb = OFF_FLAG + par * FLAG_PAR;
  const rsrc_t rs = mk_rsrc(P.ws, WS_USED);

  if (wv == 7) {
    const unsigned l4 = (unsigned)lane * 4u;
    const unsigned want = call & 0xffffffu;
    unsigned* tmo = (unsigned*)(P.ws + OFF_TMO);
    const unsigned my_entry = 1u | stage_off(0, lane >> 3, lane & 7);
    u64 todo = ~0ull;
    int qn = 0;
    auto poll = [&]() {
      if ((todo >> lane) & 1ull) {
        dma_poll2(lds0 + LDS_SNAP, l4, rs, flagb + (unsigned)xb0 * 256u);
        dma_poll1(lds0 + LDS_SNAP + 512, l4, rs, flagb + (unsigned)(16 + cb0) * 256u);
      }
    };
    auto push = [&](u64 m, unsigned ex) {
      const int pos = qn + __builtin_amdgcn_mbcnt_hi((unsigned)(m >> 32), __builtin_amdgcn_mbcnt_lo((unsigned)m, 0u));
      if ((m >> lane) & 1ull) { exps[pos] = ex; queue[pos] = my_entry; }
      todo &= ~m;
      qn += __builtin_popcountll(m);
    };
    auto plan = [&]() {
      const unsigned f0 = snap[lane], f1 = snap[64 + lane], f2 = snap[128 + lane];
      const u64 m = __ballot((f0 >> 8) == want && (f1 >> 8) == want && (f2 >> 8) == want) & todo;
      if (m) {
        const int ew = (signed char)f2;
        push(m, (unsigned)(((signed char)f0 + ew) & 0xffff) | (unsigned)((signed char)f1 + ew) << 16);
      }
    };
    for (int d = 0; d < NSTEP; ++d) {
      if (qn <= d) {
        const u64 t0 = now();
        while (true) {
          poll();
          wait_vm<POLL_AHEAD * 3>();
          plan();
          if (qn > d) break;
          const u64 dt = now() - t0;
          if (dt > 100000ull) {
            const unsigned tv = uni(__hip_atomic_load(tmo, __ATOMIC_RELAXED, __HIP_MEMORY_SCOPE_AGENT));
            if (tv == call || dt > DEADLINE_TICKS) {
              __hip_atomic_store(tmo, call, __ATOMIC_RELAXED, __HIP_MEMORY_SCOPE_AGENT);
              *bad = 1;
              push(todo, 0u);
              break;
            }
          }
          __builtin_amdgcn_s_sleep(POLL_SLEEP);
        }
        if (todo) { __builtin_amdgcn_s_sleep(POLL_LATE); poll(); }
      }
      BARRIER();
      if (todo) { plan(); if (todo) poll(); }
    }
    BARRIER();
    return;
  }
  if (wv >= 4) {
    const int pi = wv - 4;
    const int myB = pi < 2 ? xb0 + pi : 16 + cb0;
    const unsigned l16 = (unsigned)lane * 16u;
    const unsigned src0 = stage + (unsigned)myB * PIECE - 1u;
    const unsigned dst0 = lds0 + (unsigned)pi * PIECE, dst_end = dst0 + NS * STAGE_B;
    unsigned dst = dst0;
    int issued = 0, avail = 0;
    unsigned qv = 0;
    auto refresh = [&]() {
      qv = queue[lane];
      const u64 m = ~__ballot(qv != 0u);
      avail = m == 0ull ? 64 : __builtin_ctzll(m);
    };
    auto issue1 = [&]() {
      dma8(dst, l16, rs, src0 + __builtin_amdgcn_readlane(qv, issued));
      ++issued;
      dst += STAGE_B;
      if (dst == dst_end) dst = dst0;
    };
    for (int d = 0; d < NSTEP; ++d) {
      if (issued <= d) {
        while (true) { refresh(); if (avail > d) break; __builtin_amdgcn_s_sleep(1); }
        const int upto = avail < d + NS - 1 ? avail : d + NS - 1;
        while (issued < upto) issue1();
      }
      const int n = issued - d;
      wait_vm<32>();
      if (n < 5) { wait_vm<24>(); if (n < 4) { wait_vm<16>(); if (n < 3) { wait_vm<8>(); if (n < 2) wait_vm<0>(); } } }
      BARRIER();
      if (avail < d + NS && avail < NSTEP) refresh();
      const int upto = avail < d + NS ? avail : d + NS;
      while (issued < upto) issue1();
    }
    BARRIER();
    return;
  }

  const lds_ptr a_rd = smem + wv * 4096 + lane * 16;
  const lds_ptr b_rd = smem + 2 * PIECE + lane * 16;
  f4 acc[MF][4];
#pragma unroll
  for (int m = 0; m < MF; ++m)
#pragma unroll
    for (int i = 0; i < 4; ++i) acc[m][i] = f4{0.f, 0.f, 0.f, 0.f};
  int so = 0;
  auto mma = [&](const h8 (&af)[MF][2], const h8 (&bf)[2][4]) {
#pragma unroll
    for (int kb = 0; kb < 2; ++kb)
#pragma unroll
      for (int m = 0; m < MF; ++m)
#pragma unroll
        for (int i = 0; i < 4; ++i)
          acc[m][i] = __builtin_amdgcn_mfma_f32_16x16x32_f16(af[m][kb], bf[kb][i], acc[m][i], 0, 0, 0);
  };
  auto mma_scaled = [&](const h8 (&af)[MF][2], const h8 (&bf)[2][4], int e) {
#pragma unroll
    for (int m = 0; m < MF; ++m)
#pragma unroll
      for (int i = 0; i < 4; ++i) {
        f4 t = __builtin_amdgcn_mfma_f32_16x16x32_f16(af[m][0], bf[0][i], f4{0.f, 0.f, 0.f, 0.f}, 0, 0, 0);
        t = __builtin_amdgcn_mfma_f32_16x16x32_f16(af[m][1], bf[1][i], t, 0, 0, 0);
#pragma unroll
        for (int r = 0; r < 4; ++r) acc[m][i][r] += __builtin_amdgcn_ldexpf(t[r], e);
      }
  };
  const volatile LAS short* ex_rd = (const volatile LAS short*)(smem + LDS_EXPS) + (wv >> 1);
  int ev = 0;
  auto rd = [&](h8 (&an)[MF][2], h8 (&bn)[2][4]) {
    ev = *ex_rd;
    ex_rd += 2;
#pragma unroll
    for (int kb = 0; kb < 2; ++kb) {
#pragma unroll
      for (int m = 0; m < MF; ++m) an[m][kb] = *(const LAS h8*)(a_rd + so + m * 2048 + kb * 1024);
#pragma unroll
      for (int i = 0; i < 4; ++i) bn[kb][i] = *(const LAS h8*)(b_rd + so + (kb * 4 + i) * 1024);
    }
    so = (so + STAGE_B == NS * STAGE_B) ? 0 : so + STAGE_B;
  };
  auto mix = [&]() {
#pragma unroll
    for (int i = 0; i < 2 * MF + 9; ++i) {
      __builtin_amdgcn_sched_group_barrier(0x100, 1, 0);
      __builtin_amdgcn_sched_group_barrier(0x8, 1, 0);
    }
    __builtin_amdgcn_sched_group_barrier(0x8, 8 * MF - (2 * MF + 9), 0);
  };
  const unsigned row = (unsigned)(wv * 32 + (lane >> 4) * 4);
  const rsrc_t ro = remote ? rs : mk_rsrc(P.out, SHARD_BYTES);
  const unsigned ooff = remote ? OFF_BOX + (unsigned)(1024 + a * 128 + (row >> 3) * 8 + (cb0 >> 2)) * PIECE + (row & 7) * 1024u +
                                     (unsigned)((cb0 & 3) * 256 + (lane & 15) * 16)
                               : ((unsigned)(a * 128) + row) * 16384u + (unsigned)((rank >> 2) * 2048 + cb0 * 64 + (lane & 15) * 4) * 4u;
  const unsigned rstride = remote ? 1024u : 16384u, mstride = remote ? 16u * PIECE : 16u * 16384u;
  h8 a0[MF][2], b0[2][4], a1[MF][2], b1[2][4];
#define RARE(c) __builtin_expect_with_probability((c), 0, 1.0)
  BARRIER();
  rd(a0, b0);
  for (int d = 1; d < NSTEP - 1; d += 2) {
    SYNC();
    int e = (int)uni((unsigned)ev);
    if (RARE(e != 0)) { rd(a1, b1); mma_scaled(a0, b0, e); } else { rd(a1, b1); mma(a0, b0); mix(); }
    SYNC();
    e = (int)uni((unsigned)ev);
    if (RARE(e != 0)) { rd(a0, b0); mma_scaled(a1, b1, e); } else { rd(a0, b0); mma(a1, b1); mix(); }
  }
  {
    const int e = (int)uni((unsigned)ev);
    if (RARE(e != 0)) mma_scaled(a0, b0, e); else mma(a0, b0);
  }
  SYNC();
  rd(a1, b1);
  const unsigned isbad = *bad;
  {
    const int e = (int)uni((unsigned)ev);
    if (RARE(e != 0)) mma_scaled(a1, b1, e); else mma(a1, b1);
  }
  if (RARE(uni(isbad) != 0u)) {
    const float qn_ = __builtin_nanf("");
#pragma unroll
    for (int m = 0; m < MF; ++m)
#pragma unroll
      for (int i = 0; i < 4; ++i) acc[m][i] = f4{qn_, qn_, qn_, qn_};
  }
#pragma unroll
  for (int m = 0; m < MF; ++m)
#pragma unroll
    for (int r = 0; r < 4; ++r) {
      const f2 lo = gelu2(f2{acc[m][0][r], acc[m][1][r]}), hi = gelu2(f2{acc[m][2][r], acc[m][3][r]});
      const f4 v = {lo[0], lo[1], hi[0], hi[1]};
      __builtin_amdgcn_raw_buffer_store_b128(__builtin_bit_cast(u4, v), ro, ooff + (unsigned)m * mstride + (unsigned)r * rstride, 0, AUX_DEV);
    }
  if (remote) WAIT_VM0();
  BARRIER();
  if (remote && threadIdx.x == 0)
    __builtin_amdgcn_raw_buffer_store_b32(call << 8, rs, 0, OFF_BOXFLAG + (unsigned)(1024 + a * 32 + cb0) * 4u, AUX_DEV);
}

__device__ __forceinline__ void final_wait(const Params& P, int rank, int wv, int lane) {
  const unsigned call = get_call(P, rank, wv);
  const unsigned flagb = OFF_FLAG + (call & 1u) * FLAG_PAR;
  const rsrc_t rl = mk_rsrc(P.ws, WS_USED);
  unsigned* tmo = (unsigned*)(P.ws + OFF_TMO);
  const unsigned want = call << 8;
  const u64 t0 = now();
  while (true) {
    bool ok = true;
    ok = __builtin_amdgcn_raw_buffer_load_b32(rl, (unsigned)(lane & 31) * 4u, flagb + FLAG_OUT, AUX_SYS) == want;
    if (__all(ok)) break;
    const u64 dt = now() - t0;
    if (dt > 100000ull) {
      const unsigned tv = __builtin_amdgcn_readfirstlane(__hip_atomic_load(tmo, __ATOMIC_RELAXED, __HIP_MEMORY_SCOPE_AGENT));
      if (tv == call || dt > DEADLINE_TICKS) {
        __hip_atomic_store(tmo, call, __ATOMIC_RELAXED, __HIP_MEMORY_SCOPE_AGENT);
        P.out[(rank >> 2) * 2048] = __builtin_nanf("");
        for (int i = lane; i < 512; i += 64)
          P.out[(size_t)((i >> 3) * 8) * 4096 + ((rank >> 2) ^ 1) * 2048 + (i & 7) * 256] = __builtin_nanf("");
        break;
      }
    }
    asm volatile("" ::: "memory");
    if (dt < 5000ull) __builtin_amdgcn_s_sleep(32); else __builtin_amdgcn_s_sleep(FINAL_SLEEP);
  }
}

}

__global__ void __launch_bounds__(512, 2) fused_gemm_rs(Params P) {
  __shared__ __attribute__((aligned(1024))) char smem_arr[LDS_TOTAL];
  const lds_ptr smem = (lds_ptr)smem_arr;
  const int wv = __builtin_amdgcn_readfirstlane(threadIdx.x >> 6);
  const int lane = threadIdx.x & 63;
  const int xcd = blockIdx.x & 7, slot = blockIdx.x >> 3;
  static_assert(NOPEN == 4 && NGEMM == 224, "opening map");
  const int g = slot < SLOT_REM ? 0 : slot - SLOT_REM;
  const bool opens = slot >= SLOT_REM && wv == 0;
  const int op_link = g >> 2, op_oct = g & 3;
  asm volatile("" :: "s"(P.x), "s"(P.w), "s"(P.ws), "s"(P.rank), "s"(P.link[op_link].list[op_oct]), "s"(P.link[op_link].ws));
  const int rank = P.rank & 7;
  unsigned call;
  const int box = (wv - 1) * NGEMM + ((int)blockIdx.x - SLOT_REM * 8);
  if (__builtin_expect(slot >= SLOT_REM && box < 1024, 1)) {
    if (__builtin_expect(!opens, 0)) {
      const unsigned delay = (unsigned)((CONV_DELAYS >> ((box >> 7) * 8)) & 0xffull) * 25u;
      const u64 ts = now();
      while (now() - ts < delay) __builtin_amdgcn_s_sleep(4);
    }
    call = convert(P, rank, opens, P.link[op_link], P.link[op_link].list[op_oct], (unsigned)(xcd + rank) & 7u, box, wv, lane);
  } else if (slot < SLOT_REM) {
    if (wv < 2) {
      const int dd = 1 + slot * 2 + wv;
      if (dd < 8) pusher(P, rank, smem, dd, xcd, wv, lane);
      else if (blockIdx.x == 24) final_wait(P, rank, wv, lane);
    }
    return;
  } else {
    call = get_call(P, rank, wv);
  }
  const bool first_q = (xcd >> 1) == (rank & 3);
  bool remote = slot < SLOT_OWN;
  int a, cb;
  if (remote) { const int s = slot - SLOT_REM; a = s >> 2; cb = xcd * 4 + (s & 3); }
  else if (!first_q) { const int s = slot - SLOT_OWN; a = s >> 2; cb = xcd * 4 + (s & 3); }
  else { a = 3; cb = ((rank & 3) * 8 + 8 + (xcd & 1) * 12 + (slot - SLOT_OWN)) & 31; }
  const int ntask = (remote && first_q) ? 2 : 1;
  for (int task = 0; task < ntask; ++task) {
    gemm_tile(P, rank, smem, a, cb, remote, call, wv, lane);
    remote = false;
  }
}

static const char* const kOrder[2] = {
    "H00 W00 H10 W01 W10 W02 H20 W03 W20 H30 W11 F61 W30 H40 F62 W40 W12 F63 H50 W21 F70 W50 H60 F71 W13 W60 F72 H70 "
    "W31 F73 W70 W41 W22 W51 W32 W61 W23 W71 W42 W52 W62 W72 W33 V46 V66 V47 V67 O00 O10 O20 O30 O40 O50 O60 O70",
    "R60 R70 H00 W00 W01 W02 W03 H10 W10 H20 W20 W11 H60 W12 H30 W30 W13 H70 W21 H40 W40 W22 W23 H50 W50 X60 W31 W32 "
    "O00 W33 W41 O10 W42 W51 O20 W43 W52 O30 W53 O40 O50 O60 O70 Y00 Y10 Y20 Y30 Y01 Y11 Y21 Y31 Y02 Y12 Y22 Y32 Z06 "
    "Z16 Z07 Z17"};

static void build_lists(Params& P, int rank) {
  for (int dd = 1; dd < NRANK; ++dd) {
    const int cls = dd >> 2, dest = rank ^ dd, G = dest >> 2, q = dest & 3;
    const char* s = kOrder[cls];
    int n = 0;
    while (*s && n < NOCT) {
      if (*s == ' ') { ++s; continue; }
      const char c = s[0];
      const int t = s[1] - '0', sub = s[2] - '0';
      s += 3;
      unsigned src, kind = 0, dst, flag, dep;
      auto staged = [&](int B, int from) {
        dst = (OFF_STAGE + (unsigned)((from * 8 + t) * 48 + B) * PIECE) / 1024u;
        flag = (unsigned)(B * 64 + t * 8 + from);
      };
      auto boxed = [&](int blk) { src = OFF_BOX / PIECE + (unsigned)(t * 128 + blk); dep = OFF_BOXFLAG / 4u + (unsigned)(t * 128 + blk); };
      if (c == 'W' || c == 'R') {
        const int cb = ((sub + q) & 3) * 8;
        boxed(G * 32 + cb); staged(16 + cb, rank);
      } else if (c == 'X') {
        if (dd == 4) continue;
        const int cb = (rank & 3) * 8;
        boxed(G * 32 + cb); staged(16 + cb, rank);
      } else if (c == 'V') {
        const int cb = ((sub + 2 * q) & 7) * 4;
        boxed(G * 32 + cb); staged(16 + cb, rank); kind = 5;
      } else if (c == 'Z') {
        const int rg = t * 8, cg = (sub + 2 * rank) & 7;
        src = OFF_BOX / PIECE + (unsigned)(1024 + q * 128 + rg * 8 + cg);
        dst = (unsigned)((rank & 3) * 128 + rg * 8) * 16u + (unsigned)((rank >> 2) * 8 + cg);
        flag = 0;
        dep = OFF_BOXFLAG / 4u + 1024u + (unsigned)(q * 32 + cg * 4);
        kind = 2;
      } else if (c == 'H') {
        boxed(64 + 32 * (G ^ 1) + 2 * q); staged(8, rank); kind = 1;
      } else if (c == 'O') {
        boxed(64 + 8 * dest); staged(0, rank);
      } else if (c == 'F') {
        staged(16 + (rank & 3) * 8, (G ^ 1) * 4 + ((sub + rank) & 3));
        src = dst / 8u; dep = OFF_FLAG / 4u + flag; kind = 4;
      } else {
        const int rg = t * 4, cg = ((sub + rank) & 3) * 2;
        src = OFF_BOX / PIECE + (unsigned)(1024 + q * 128 + rg * 8 + cg);
        dst = (unsigned)((rank & 3) * 128 + rg * 8) * 16u + (unsigned)((rank >> 2) * 8 + cg);
        flag = 0;
        dep = OFF_BOXFLAG / 4u + 1024u + (unsigned)(q * 32 + cg * 4);
        kind = 3;
      }
      P.link[dd - 1].list[n][0] = src | kind << 13 | dst << 16;
      P.link[dd - 1].list[n][1] = flag | dep << 16;
      ++n;
    }
    P.nitem[dd - 1] = n * 8;
    for (; n < NOCT; ++n) P.link[dd - 1].list[n] = P.link[dd - 1].list[0];
  }
}

extern "C" void kernel_launch(int rank, int nranks, ncclComm_t comm,
                              hipStream_t stream,
                              void* const* d_in, const int* in_sizes, int n_in,
                              void* d_out, int out_size,
                              const int* in_dtypes, int out_dtype,
                              void* const* d_out_all,
                              void* const* d_ws_all, size_t ws_bytes) {
  (void)comm; (void)in_dtypes; (void)out_dtype;
  if (nranks != NRANK || n_in != 2 || in_sizes[0] != 4096 * 512 || in_sizes[1] != 512 * 4096 ||
      out_size != 512 * 4096 || ws_bytes < WS_USED || (unsigned)rank >= (unsigned)NRANK) {
    fprintf(stderr, "kernel_launch: unexpected problem shape\n");
    return;
  }
  Params P;
  P.x = (const float*)d_in[0];
  P.w = (const float*)d_in[1];
  P.ws = (char*)d_ws_all[rank];
  P.out = (float*)d_out;
  P.rank = rank;
  for (int dd = 1; dd < NRANK; ++dd) { P.link[dd - 1].ws = (char*)d_ws_all[rank ^ dd]; P.link[dd - 1].out = (float*)d_out_all[rank ^ dd]; }
  build_lists(P, rank);
  fused_gemm_rs<<<256, 512, 0, stream>>>(P);
  HIP_CHECK(hipGetLastError());
}
```
